```python
import numpy as np

kernel_name = "gpu_gemm_a2a_m4096_k4096_n4096_f32_silu"


ABSMAX_RTOL = 0.001

COMPUTE_DTYPE = "float32"
EPILOGUE = "silu"
IN_QUANT = None
OUT_QUANT = None
IN_DTYPES = {"x": "float32", "w_mat": "float32"}
WORLD_SIZE = 8

def setup_inputs():

    x = np.random.randn(4096, 4096).astype(np.float32)
    w_mat = (np.random.randn(4096, 4096) / np.sqrt(4096)).astype(np.float32)
    return {"x": x, "w_mat": w_mat}

def _epilogue(y):
    return (y / (1.0 + np.exp(-np.clip(y, -60.0, 60.0)))).astype(np.float32)

def reference(x, w_mat):
    y = (x @ w_mat).astype(np.float32)
    return _epilogue(y)
```

```cpp
#include <hip/hip_runtime.h>
#include <rccl/rccl.h>
#include <cstdio>
#include <cstdint>

#define HIP_CHECK(expr) do { hipError_t _e = (expr); if (_e != hipSuccess) { \
    fprintf(stderr, "HIP error %s at %s:%d\n", hipGetErrorString(_e), __FILE__, __LINE__); \
    return; } } while (0)

typedef _Float16 h8 __attribute__((ext_vector_type(8)));
typedef _Float16 h4 __attribute__((ext_vector_type(4)));
typedef _Float16 h2 __attribute__((ext_vector_type(2)));
typedef float f4 __attribute__((ext_vector_type(4)));
typedef float f2 __attribute__((ext_vector_type(2)));

constexpr int NR = 8;
constexpr int ML = 512;
constexpr int KK = 4096;
constexpr int NN = 4096;
constexpr int NB = 512;
constexpr int BM = 128, BN = 64, BK = 128;
constexpr int NWG = 256;
constexpr unsigned long long DEADLINE_TICKS = 50000000ull;

constexpr size_t WS_TICKET = 0;
constexpr size_t WS_TMO = 128;
constexpr size_t WS_FLAGS = 256;
constexpr size_t WS_USED = WS_FLAGS + NR * 128;

struct Params {
    const float* x;
    const float* w;
    float* out[NR];
    unsigned char* ws[NR];
    int rank;
};

__device__ __forceinline__ float silu(float y) {
    float c = fminf(fmaxf(y, -60.0f), 60.0f);
    return y / (1.0f + __expf(-c));
}

constexpr int LDA = BK + 8;

__global__ __launch_bounds__(256, 1) void fused_kernel(Params p) {
    __shared__ __attribute__((aligned(16))) unsigned char smem[131072];
    __shared__ unsigned s_call;
    _Float16* As = (_Float16*)smem;
    const int tid = threadIdx.x, lane = tid & 63, wave = tid >> 6;
    const int bid = blockIdx.x;
    const int rank = p.rank;
    const int xcd = bid & 7, idx = bid >> 3;
    const int dest = (xcd + rank) & 7;
    const int mi = idx & 3, ni = idx >> 2;
    const int row0 = mi * BM;
    const int col0 = dest * NB + ni * BN;

    unsigned char* myws = p.ws[rank];
    unsigned ticket = 0;
    if (tid == 0) ticket = __hip_atomic_fetch_add((unsigned*)(myws + WS_TICKET), 1u, __ATOMIC_RELAXED, __HIP_MEMORY_SCOPE_AGENT);

    const float* xg = p.x + (size_t)(row0 + (tid >> 5)) * KK + (tid & 31) * 4;
    const int c = lane & 15, g = lane >> 4;
    const float* wg = p.w + (size_t)(wave * 32 + g * 8) * NN + col0 + c * 4;

    f4 acc[8][4];
#pragma unroll
    for (int i = 0; i < 8; ++i)
#pragma unroll
        for (int t = 0; t < 4; ++t) acc[i][t] = (f4){0.f, 0.f, 0.f, 0.f};

    f4 xr[16], wr[8];
#pragma unroll
    for (int i = 0; i < 16; ++i) xr[i] = *(const f4*)(xg + (size_t)i * 8 * KK);
#pragma unroll
    for (int j = 0; j < 8; ++j) wr[j] = *(const f4*)(wg + (size_t)j * NN);

    constexpr int NT = KK / BK;
    for (int kt = 0; kt < NT; ++kt) {
        _Float16* A = As + (kt & 1) * BM * LDA;
#pragma unroll
        for (int i = 0; i < 16; ++i) {
            h4 v = {(_Float16)xr[i][0], (_Float16)xr[i][1], (_Float16)xr[i][2], (_Float16)xr[i][3]};
            *(h4*)(A + ((tid >> 5) + 8 * i) * LDA + (tid & 31) * 4) = v;
        }
        h8 bf[4];
#pragma unroll
        for (int t = 0; t < 4; ++t)
#pragma unroll
            for (int j = 0; j < 8; ++j) bf[t][j] = (_Float16)wr[j][t];
        if (kt + 1 < NT) {
            const float* xn = xg + (size_t)(kt + 1) * BK;
            const float* wn = wg + (size_t)(kt + 1) * BK * NN;
#pragma unroll
            for (int i = 0; i < 16; ++i) xr[i] = *(const f4*)(xn + (size_t)i * 8 * KK);
#pragma unroll
            for (int j = 0; j < 8; ++j) wr[j] = *(const f4*)(wn + (size_t)j * NN);
        }
        __syncthreads();
#pragma unroll
        for (int i = 0; i < 8; ++i) {
            h8 af = *(const h8*)(A + (i * 16 + c) * LDA + wave * 32 + g * 8);
#pragma unroll
            for (int t = 0; t < 4; ++t) acc[i][t] = __builtin_amdgcn_mfma_f32_16x16x32_f16(af, bf[t], acc[i][t], 0, 0, 0);
        }
    }
    __syncthreads();
    f4* red = (f4*)smem;
#pragma unroll
    for (int i = 0; i < 8; ++i)
#pragma unroll
        for (int j = 0; j < 4; ++j) {
            f4 v = {acc[i][0][j], acc[i][1][j], acc[i][2][j], acc[i][3][j]};
            red[((wave * 8 + i) * 4 + j) * 64 + lane] = v;
        }
    __syncthreads();
    if (tid == 0) s_call = ticket / NWG + 1;
    float* outp = p.out[dest] + (size_t)(rank * ML + row0) * NB + ni * BN + c * 4;
#pragma unroll
    for (int ii = 0; ii < 2; ++ii) {
        const int i = wave * 2 + ii;
#pragma unroll
        for (int j = 0; j < 4; ++j) {
            f4 v = red[((0 * 8 + i) * 4 + j) * 64 + lane];
#pragma unroll
            for (int w = 1; w < 4; ++w) v += red[((w * 8 + i) * 4 + j) * 64 + lane];
            f4 o = {silu(v[0]), silu(v[1]), silu(v[2]), silu(v[3])};
            *(f4*)(outp + (size_t)(i * 16 + g * 4 + j) * NB) = o;
        }
    }
    asm volatile("s_waitcnt vmcnt(0)" ::: "memory");
    __syncthreads();
    if (tid == 0) {
        const unsigned call = s_call;
        if (dest != rank) {
            __builtin_amdgcn_fence(__ATOMIC_RELEASE, "");
            asm volatile("s_waitcnt vmcnt(0)" ::: "memory");
            __hip_atomic_fetch_add((unsigned*)(p.ws[dest] + WS_FLAGS + rank * 128), 1u, __ATOMIC_RELAXED, __HIP_MEMORY_SCOPE_SYSTEM);
        }
        if (dest == rank && idx < NR && idx != rank) {
            unsigned* flag = (unsigned*)(myws + WS_FLAGS + idx * 128);
            unsigned* tmo = (unsigned*)(myws + WS_TMO);
            const unsigned target = call * 32u;
            unsigned long long t0 = __builtin_amdgcn_s_memrealtime();
            while (__hip_atomic_load(flag, __ATOMIC_RELAXED, __HIP_MEMORY_SCOPE_SYSTEM) < target) {
                if (__hip_atomic_load(tmo, __ATOMIC_RELAXED, __HIP_MEMORY_SCOPE_AGENT) == call ||
                    __builtin_amdgcn_s_memrealtime() - t0 > DEADLINE_TICKS) {
                    __hip_atomic_store(tmo, call, __ATOMIC_RELAXED, __HIP_MEMORY_SCOPE_AGENT);
                    outp[0] = __builtin_nanf("");
                    break;
                }
                __builtin_amdgcn_s_sleep(2);
            }
            __builtin_amdgcn_fence(__ATOMIC_ACQUIRE, "");
        }
    }
}

extern "C" void kernel_launch(int rank, int nranks, ncclComm_t comm,
                              hipStream_t stream,
                              void* const* d_in, const int* in_sizes, int n_in,
                              void* d_out, int out_size,
                              const int* in_dtypes, int out_dtype,
                              void* const* d_out_all,
                              void* const* d_ws_all, size_t ws_bytes) {
    if (nranks != NR || ws_bytes < WS_USED || n_in < 2) { fprintf(stderr, "bad config\n"); return; }
    Params p;
    p.x = (const float*)d_in[0];
    p.w = (const float*)d_in[1];
    for (int i = 0; i < NR; ++i) { p.out[i] = (float*)d_out_all[i]; p.ws[i] = (unsigned char*)d_ws_all[i]; }
    p.rank = rank;
    fused_kernel<<<NWG, 256, 0, stream>>>(p);
    HIP_CHECK(hipGetLastError());
}
```

```cpp
#include <hip/hip_runtime.h>
#include <rccl/rccl.h>
#include <cstdio>
#include <cstdint>

#define HIP_CHECK(expr) do { hipError_t _e = (expr); if (_e != hipSuccess) { \
    fprintf(stderr, "HIP error %s at %s:%d\n", hipGetErrorString(_e), __FILE__, __LINE__); \
    return; } } while (0)

typedef _Float16 h8 __attribute__((ext_vector_type(8)));
typedef float f4 __attribute__((ext_vector_type(4)));
typedef unsigned u4 __attribute__((ext_vector_type(4)));

#define RING_N 2
#define WWINB 4
#define APOL 0
#define RPOL 16
#define WPOL 2
#define XPOL 2
#define PUSHN 2
#define NWB 2
#define RSLEEP 2
#define XPF 0
#define XPFD 3
#define ORDER 1

constexpr int NR = 8;
constexpr int ML = 512;
constexpr int KK = 4096;
constexpr int NN = 4096;
constexpr int NB = 512;
constexpr int NWG = 256;
constexpr int NTH = 512;
constexpr int NPASS = 16;
constexpr int RING = RING_N;
constexpr int ORING = 8;
constexpr unsigned SLOT_BYTES = 4u << 20;
#define DLT 50000000ull
constexpr unsigned long long DEADLINE_TICKS = DLT;
static_assert(NPASS % RING == 0 && (NPASS / RING) % 2 == 0, "tag parity assumes an even number of uses per slot and call");

constexpr unsigned OFF_TMO = 128;
constexpr unsigned OFF_RFLAG = 256;
constexpr unsigned OFF_XRDY = 2048;
constexpr unsigned OFF_CTR = 3072;
constexpr unsigned OFF_XCC = 4096;
constexpr unsigned OFF_DONE = 8192;
constexpr unsigned OFF_X16 = 1u << 20;
constexpr unsigned OFF_PART = 9u << 20;
constexpr size_t WS_USED = (size_t)OFF_PART + (size_t)RING * SLOT_BYTES;
constexpr unsigned OFF_DBG = 48u << 20;

constexpr int LDS_W = 0;
constexpr int LDS_OUT = 131072;
constexpr int LDS_FLAG = LDS_OUT + ORING * 2048;
constexpr int LDS_BYTES = LDS_FLAG + 256;
enum { F_CALL = 0, F_XCNT, F_WA, F_WB, F_CDONE, F_RTICKET, F_OFREE, F_SAME, F_XALL, F_ORDY  , F_NWORDS = F_ORDY + 16 };

struct Params {
    const float* x;
    const float* w;
    float* out[NR];
    unsigned char* ws[NR];
    int rank;
};

#define STAMPOK(i) true
#define STAMP(i) do {} while (0)

__device__ __forceinline__ float silu(float y) {
    float c = fminf(fmaxf(y, -60.0f), 60.0f);
    return y * __builtin_amdgcn_rcpf(1.0f + __expf(-c));
}

typedef __attribute__((address_space(3))) unsigned lds_u32;
__device__ __forceinline__ unsigned lds_load(lds_u32* p) { return __hip_atomic_load(p, __ATOMIC_RELAXED, __HIP_MEMORY_SCOPE_WORKGROUP); }
__device__ __forceinline__ void lds_add1(lds_u32* p, int lane) {
    asm volatile("s_waitcnt lgkmcnt(0)" ::: "memory");
    if (lane == 0) __hip_atomic_fetch_add(p, 1u, __ATOMIC_RELAXED, __HIP_MEMORY_SCOPE_WORKGROUP);
}

__device__ __forceinline__ unsigned lds_ticket(lds_u32* p) {
    unsigned old;
    unsigned long long save;
    const unsigned addr = (unsigned)(size_t)p, one = 1u;
    asm volatile("s_mov_b64 %1, exec\n\ts_mov_b64 exec, 1\n\tds_add_rtn_u32 %0, %2, %3\n\ts_waitcnt lgkmcnt(0)\n\ts_mov_b64 exec, %1"
                 : "=&v"(old), "=&s"(save) : "v"(addr), "v"(one) : "memory");
    return __builtin_amdgcn_readfirstlane(old);
}

__device__ __forceinline__ bool lds_wait_ge(lds_u32* p, unsigned target) {
    if ((unsigned)__builtin_amdgcn_readfirstlane(lds_load(p)) >= target) return true;
    unsigned long long t0 = __builtin_amdgcn_s_memrealtime();
    while ((unsigned)__builtin_amdgcn_readfirstlane(lds_load(p)) < target) {
        if (__builtin_amdgcn_s_memrealtime() - t0 > 2 * DEADLINE_TICKS) return false;
        __builtin_amdgcn_s_sleep(1);
    }
    return true;
}

__device__ __forceinline__ bool give_up(unsigned long long t0, unsigned* tmo, unsigned call) {
    if ((unsigned)__builtin_amdgcn_readfirstlane(__hip_atomic_load(tmo, __ATOMIC_RELAXED, __HIP_MEMORY_SCOPE_AGENT)) == call ||
        __builtin_amdgcn_s_memrealtime() - t0 > DEADLINE_TICKS) {
        __hip_atomic_store(tmo, call, __ATOMIC_RELAXED, __HIP_MEMORY_SCOPE_AGENT);
        return true;
    }
    return false;
}

__device__ __forceinline__ bool gwait_all_ge(unsigned* word, bool sel, unsigned target, unsigned* tmo, unsigned call) {
    unsigned long long t0 = __builtin_amdgcn_s_memrealtime();
    for (;;) {
        const unsigned v = __hip_atomic_load(word, __ATOMIC_RELAXED, __HIP_MEMORY_SCOPE_AGENT);
        if (__all(!sel || v >= target)) return true;
        if (give_up(t0, tmo, call)) return false;
        __builtin_amdgcn_s_sleep(1);
    }
}

#define RSRC(ptr, bytes) __builtin_amdgcn_make_buffer_rsrc((void*)(ptr), 0, (bytes), 0x00020000)
#define BLOAD(rs, voff, soff, aux) __builtin_amdgcn_raw_buffer_load_b128(rs, voff, soff, aux)

__device__ __forceinline__ h8 cvt8(f4 a, f4 b) {
    h8 r = {(_Float16)a[0], (_Float16)a[1], (_Float16)a[2], (_Float16)a[3], (_Float16)b[0], (_Float16)b[1], (_Float16)b[2], (_Float16)b[3]};
    return r;
}

template <int NU, int WIN>
__device__ __forceinline__ void load_w_units(__amdgpu_buffer_rsrc_t wrs, unsigned voff, unsigned soff0, unsigned char* ldsw, int lane) {
    f4 t[WIN][8];
#define W_ISSUE(u) _Pragma("unroll") for (int jj = 0; jj < 8; ++jj) \
        t[(u) % WIN][jj] = __builtin_bit_cast(f4, BLOAD(wrs, voff, soff0 + (unsigned)((u) * 32 + (jj & 3) + 16 * (jj >> 2)) * (NN * 4u), WPOL))
#pragma unroll
    for (int u = 0; u < WIN; ++u) { W_ISSUE(u); }
#pragma unroll
    for (int u = 0; u < NU; ++u) {
        h8 f[4];
#pragma unroll
        for (int tt = 0; tt < 4; ++tt)
#pragma unroll
            for (int jj = 0; jj < 8; ++jj) f[tt][jj] = (_Float16)t[u % WIN][jj][tt];
        __builtin_amdgcn_sched_barrier(0);
        if (u + WIN < NU) { W_ISSUE(u + WIN); }
#pragma unroll
        for (int tt = 0; tt < 4; ++tt) *(h8*)(ldsw + (u * 8 + tt) * 1024 + lane * 16) = f[tt];
        __builtin_amdgcn_sched_barrier(0);
    }
#undef W_ISSUE
}

__device__ __forceinline__ int pass_m(int q) { return q < 4 ? q : (q < 12 ? 4 + ((q - 4) >> 1) : 15 - q); }
__device__ __forceinline__ int pass_h(int q) { return q < 4 ? 0 : (q < 12 ? (q & 1) : 1); }
__device__ __forceinline__ int visit_m(int v) { return v < 8 ? v : 11 - v; }
constexpr int NVISIT = 12;

typedef __attribute__((address_space(3))) const h8 lds_h8;
__device__ __forceinline__ h8 lds_frag(unsigned addr, int imm) { return *(lds_h8*)(size_t)(addr + imm); }

struct Fin {
    __amdgpu_buffer_rsrc_t wsrs;
    unsigned char* myws;
    unsigned* tmo;
    lds_u32* cdone;
    unsigned pvoff, dvoff0, seq0, call;
    int lane;
    bool same;
};

__device__ __forceinline__ void publish(const Fin& f, int ps, const f4 (&acc)[4], unsigned dv) {
    if (ps >= RING) {
        const unsigned need = f.seq0 + (ps - RING + 1);
        if (!__all(dv >= need)) gwait_all_ge((unsigned*)(f.myws + f.dvoff0 + (ps % RING) * 4u), true, need, f.tmo, f.call);
    }
    const unsigned so = (unsigned)(ps % RING) * SLOT_BYTES;
    const unsigned par = ((ps / RING) + 1) & 1;
#pragma unroll
    for (int jr = 0; jr < 4; ++jr) {
        u4 v = {(__float_as_uint(acc[0][jr]) & ~1u) | par, __float_as_uint(acc[1][jr]), (__float_as_uint(acc[2][jr]) & ~1u) | par, __float_as_uint(acc[3][jr])};
        if (f.same) __builtin_amdgcn_raw_buffer_store_b128(v, f.wsrs, f.pvoff + jr * 512, so, 0);
        else __builtin_amdgcn_raw_buffer_store_b128(v, f.wsrs, f.pvoff + jr * 512, so, 16);
    }
    if (f.lane == 0) __hip_atomic_fetch_add(f.cdone, 1u, __ATOMIC_RELAXED, __HIP_MEMORY_SCOPE_WORKGROUP);
}

template <bool RELOAD, bool PREV>
__device__ __forceinline__ void pass_body(int ps, h8 (&a)[16], f4 (&acc)[4], const f4 (&accp)[4], unsigned dvp, unsigned& dv, h8 (&b)[3][4],
                                          unsigned wb, unsigned wb_next, const Fin& f, unsigned avoff, unsigned nsoff, unsigned pfoff, unsigned psoff, unsigned& pf) {
    unsigned lo = wb, hi = wb + 65536u;
    asm volatile("" : "+v"(lo), "+v"(hi));
#pragma unroll
    for (int s = 0; s < 16; ++s) {
#pragma unroll
        for (int t = 0; t < 4; ++t) {
            if (s == 0) acc[t] = __builtin_amdgcn_mfma_f32_16x16x32_f16(a[s], b[0][t], (f4){0.f, 0.f, 0.f, 0.f}, 0, 0, 0);
            else acc[t] = __builtin_amdgcn_mfma_f32_16x16x32_f16(a[s], b[s % 3][t], acc[t], 0, 0, 0);
            if (s + 2 < 8) b[(s + 2) % 3][t] = lds_frag(lo, ((s + 2) * 8 + t) * 1024);
            else if (s + 2 < 16) b[(s + 2) % 3][t] = lds_frag(hi, ((s + 2 - 8) * 8 + t) * 1024);
            __builtin_amdgcn_sched_barrier(0);
        }
        if (s == 0) {
            if (PREV) publish(f, ps - 1, accp, dvp);
            asm volatile("" ::: "memory");
            dv = __builtin_amdgcn_raw_buffer_load_b32(f.wsrs, f.dvoff0 + (ps % RING) * 4u, 0, 16);
            asm volatile("" ::: "memory");
        }
        if (RELOAD) a[s] = __builtin_bit_cast(h8, BLOAD(f.wsrs, avoff + (s & 3) * 1024, nsoff + (s >> 2) * 4096, APOL));
        __builtin_amdgcn_sched_barrier(0);
    }
    unsigned nx = wb_next;
    asm volatile("" : "+v"(nx));
#pragma unroll
    for (int s = 0; s < 2; ++s)
#pragma unroll
        for (int t = 0; t < 4; ++t) b[s][t] = lds_frag(nx, (s * 8 + t) * 1024);
}

__global__ __launch_bounds__(NTH, 2) void fused_kernel(Params p) {
    __shared__ __attribute__((aligned(16))) unsigned char smem[LDS_BYTES];
    const int tid = threadIdx.x, lane = tid & 63;
    const int wave = __builtin_amdgcn_readfirstlane(tid >> 6);
    const int bid = blockIdx.x;
    const int rank = p.rank;
    const int j = bid & 7, idx = bid >> 3;
    const int dest = j;
    const int kq = idx & 7, nb = idx >> 3;
    const int grp = j * 4 + nb;
    unsigned char* myws = p.ws[rank];
    lds_u32* lf = (lds_u32*)(smem + LDS_FLAG);
    if (tid < F_NWORDS) lf[tid] = 0;
    __syncthreads();
    STAMP(0);
    unsigned* tmo = (unsigned*)(myws + OFF_TMO);
    const auto wsrs = RSRC(myws, (unsigned)WS_USED);
    const int c = lane & 15, g = lane >> 4;

    if (wave < 4) {
        unsigned ctr = 0;
        if (wave == 0) ctr = __hip_atomic_load((unsigned*)(myws + OFF_CTR) + bid, __ATOMIC_RELAXED, __HIP_MEMORY_SCOPE_AGENT);
        {
            const auto xrs = RSRC(p.x, ML * KK * 4u);
            const unsigned voff = ((unsigned)(idx * 16 + c) * KK + j * 512 + wave * 128 + g * 4) * 4u;
            f4 t[8];
#pragma unroll
            for (int q = 0; q < 4; ++q) {
                t[2 * q] = __builtin_bit_cast(f4, BLOAD(xrs, voff + q * 128, 0, XPOL));
                t[2 * q + 1] = __builtin_bit_cast(f4, BLOAD(xrs, voff + q * 128 + 64, 0, XPOL));
            }
            if (wave == 0) {
                const unsigned call0 = __builtin_amdgcn_readfirstlane(ctr) + 1;
                if (lane == 0) {
                    __hip_atomic_store(&lf[F_CALL], call0, __ATOMIC_RELAXED, __HIP_MEMORY_SCOPE_WORKGROUP);
                    unsigned xcc;
                    asm volatile("s_getreg_b32 %0, hwreg(HW_REG_XCC_ID, 0, 4)" : "=s"(xcc));
                    __hip_atomic_store((unsigned*)(myws + OFF_XCC + grp * 128 + kq * 4), (call0 << 4) | xcc, __ATOMIC_RELAXED, __HIP_MEMORY_SCOPE_AGENT);
                    __hip_atomic_store((unsigned*)(myws + OFF_CTR) + bid, call0, __ATOMIC_RELAXED, __HIP_MEMORY_SCOPE_AGENT);
                }
            }
            u4 f[4];
#pragma unroll
            for (int q = 0; q < 4; ++q) {
                f[q] = __builtin_bit_cast(u4, cvt8(t[2 * q], t[2 * q + 1]));
                if (f[q][0] == 0u) f[q][0] = 0x8000u;
                if (f[q][2] == 0u) f[q][2] = 0x8000u;
            }
            lds_wait_ge(&lf[F_CALL], 1);
            const unsigned call0 = __builtin_amdgcn_readfirstlane(lds_load(&lf[F_CALL]));
            const unsigned xo = OFF_X16 + (call0 & 1u) * (4u << 20) + (unsigned)((idx * 8 + j) * 16 + wave * 4) * 1024u + lane * 16u;
#pragma unroll
            for (int q = 0; q < 4; ++q) __builtin_amdgcn_raw_buffer_store_b128(f[q], wsrs, xo + q * 1024, 0, 16);
        }
        STAMP(1);
        const unsigned call = __builtin_amdgcn_readfirstlane(lds_load(&lf[F_CALL]));
        const unsigned avoff = OFF_X16 + (call & 1u) * (4u << 20) + (unsigned)((wave * 8 + kq) * 16) * 1024u + lane * 16u;
        h8 a[2][16];
        unsigned xcc_v;
        bool xall;
        {
            asm volatile("s_waitcnt vmcnt(0)" ::: "memory");
            if (lds_ticket(&lf[F_XCNT]) == 3) {
                if (lane == 0) __hip_atomic_store((unsigned*)(myws + OFF_XRDY + j * 128) + idx, call, __ATOMIC_RELAXED, __HIP_MEMORY_SCOPE_AGENT);
            }
            unsigned* tab = (unsigned*)(myws + OFF_XCC + grp * 128) + (lane & 7);
            unsigned long long t0 = __builtin_amdgcn_s_memrealtime();
            bool need0 = true, need1 = true;
            for (;;) {
                asm volatile("" ::: "memory");
                xcc_v = __hip_atomic_load(tab, __ATOMIC_RELAXED, __HIP_MEMORY_SCOPE_AGENT);
                const unsigned xr_v = __hip_atomic_load((unsigned*)(myws + OFF_XRDY + kq * 128) + (lane & 31), __ATOMIC_RELAXED, __HIP_MEMORY_SCOPE_AGENT);
                __builtin_amdgcn_sched_barrier(0);
                if (need0) {
#pragma unroll
                    for (int s = 0; s < 16; ++s) a[0][s] = __builtin_bit_cast(h8, BLOAD(wsrs, avoff + (s & 3) * 1024, (s >> 2) * 4096, 16));
                }
                if (need1) {
#pragma unroll
                    for (int s = 0; s < 16; ++s) a[1][s] = __builtin_bit_cast(h8, BLOAD(wsrs, avoff + (s & 3) * 1024, (512u << 10) + (s >> 2) * 4096, 16));
                }
                __builtin_amdgcn_sched_barrier(0);
                xall = __all(xr_v >= call);
                unsigned m0 = ~0u, m1 = ~0u;
#pragma unroll
                for (int s = 0; s < 16; ++s) {
                    const u4 w0 = __builtin_bit_cast(u4, a[0][s]), w1 = __builtin_bit_cast(u4, a[1][s]);
                    m0 = min(m0, min(w0[0], w0[2]));
                    m1 = min(m1, min(w1[0], w1[2]));
                }
                need0 = !__all(m0 != 0u);
                need1 = !__all(m1 != 0u);
                if (!need0 && !need1 && __all((xcc_v >> 4) >= call)) break;
                if (give_up(t0, tmo, call)) break;
            }
        }
        STAMP(2);
        const unsigned wb0 = (unsigned)(size_t)(lds_h8*)(smem + LDS_W) + lane * 16;
        Fin f;
        f.wsrs = wsrs; f.myws = myws; f.tmo = tmo; f.cdone = &lf[F_CDONE]; f.call = call; f.lane = lane;
        f.dvoff0 = OFF_DONE + (unsigned)(grp * 8 + 2 * wave + (lane & 1)) * 128u;
        f.seq0 = (call - 1) * NPASS;
        f.pvoff = OFF_PART + (unsigned)((grp * 8 + 2 * wave + (lane >> 5)) * 8 + kq) * 2048u + (lane & 31) * 16u;
        f.same = __all(xcc_v == (unsigned)__builtin_amdgcn_readfirstlane(xcc_v));
        if (wave == 0 && lane == 0) __hip_atomic_store(&lf[F_SAME], f.same ? 1u : 2u, __ATOMIC_RELAXED, __HIP_MEMORY_SCOPE_WORKGROUP);
        lds_wait_ge(&lf[F_WA], 4);
        STAMP(3);
        f4 acc[2][4];
        unsigned dv[2] = {0u, 0u};
        h8 b[3][4];
#pragma unroll
        for (int s = 0; s < 2; ++s)
#pragma unroll
            for (int t = 0; t < 4; ++t) b[s][t] = lds_frag(wb0, (s * 8 + t) * 1024);
        auto voff2 = [&](int v) { return (v + 2 < NVISIT) ? (unsigned)visit_m(v + 2) * (512u << 10) : 0x80000000u; };
        auto voffp = [&](int v) { return (v + XPFD < NVISIT) ? (unsigned)visit_m(v + XPFD) * (512u << 10) : 0x80000000u; };
        const unsigned pfoff = (lane < 32) ? (avoff - lane * 16u) + (unsigned)(nb * 32 + lane) * 128u : 0x80000000u;
        unsigned pf = 0;
        const unsigned wbA = wb0, wbB = wb0 + 4096;
        pass_body<false, false>(0, a[0], acc[0], acc[1], 0u, dv[0], b, wbA, wbA, f, avoff, 0u, pfoff, 0x80000000u, pf);
        STAMP(4);
        publish(f, 0, acc[0], dv[0]);
        if (!xall) lds_wait_ge(&lf[F_XALL], 1);
#pragma unroll
        for (int s = 0; s < 16; ++s) a[0][s] = __builtin_bit_cast(h8, BLOAD(wsrs, avoff + (s & 3) * 1024, voff2(0) + (s >> 2) * 4096, APOL));
        pass_body<true, false>(1, a[1], acc[1], acc[0], 0u, dv[1], b, wbA, wbA, f, avoff, voff2(1), pfoff, voffp(1), pf);
        STAMP(5);
        pass_body<true, true>(2, a[0], acc[0], acc[1], dv[1], dv[0], b, wbA, wbA, f, avoff, voff2(2), pfoff, voffp(2), pf);
        STAMP(6);
        pass_body<true, true>(3, a[1], acc[1], acc[0], dv[0], dv[1], b, wbA, wbA, f, avoff, voff2(3), pfoff, voffp(3), pf);
        STAMP(7);
        lds_wait_ge(&lf[F_WB], NWB);
        for (int v = 4; v < 8; v += 2) {
            const int ps = 4 + 2 * (v - 4);
            pass_body<false, true>(ps, a[0], acc[0], acc[1], dv[1], dv[0], b, wbA, wbB, f, avoff, 0u, pfoff, 0u, pf);
            STAMP(4 + ps);
            pass_body<true, true>(ps + 1, a[0], acc[1], acc[0], dv[0], dv[1], b, wbB, wbA, f, avoff, voff2(v), pfoff, voffp(v), pf);
            STAMP(5 + ps);
            pass_body<false, true>(ps + 2, a[1], acc[0], acc[1], dv[1], dv[0], b, wbA, wbB, f, avoff, 0u, pfoff, 0u, pf);
            STAMP(6 + ps);
            pass_body<true, true>(ps + 3, a[1], acc[1], acc[0], dv[0], dv[1], b, wbB, v == 6 ? wbB : wbA, f, avoff, voff2(v + 1), pfoff, voffp(v + 1), pf);
            STAMP(7 + ps);
        }
        for (int v = 8; v < 12; v += 2) {
            pass_body<true, true>(v + 4, a[0], acc[0], acc[1], dv[1], dv[0], b, wbB, wbB, f, avoff, voff2(v), pfoff, voffp(v), pf);
            STAMP(8 + v);
            pass_body<true, true>(v + 5, a[1], acc[1], acc[0], dv[0], dv[1], b, wbB, wbB, f, avoff, voff2(v + 1), pfoff, voffp(v + 1), pf);
            STAMP(9 + v);
        }
        publish(f, NPASS - 1, acc[1], dv[1]);
        {
            const unsigned xz = OFF_X16 + ((call + 1) & 1u) * (4u << 20) + (unsigned)((idx * 8 + j) * 16 + wave * 4) * 1024u + lane * 16u;
            const u4 z = {0u, 0u, 0u, 0u};
#pragma unroll
            for (int q = 0; q < 4; ++q) __builtin_amdgcn_raw_buffer_store_b128(z, wsrs, xz + q * 1024, 0, 16);
        }
    } else {
        const auto wrs = RSRC(p.w, (unsigned)KK * NN * 4u);
        const unsigned wvoff = ((unsigned)(kq * 512 + g * 4) * NN + dest * NB + nb * 128 + c * 4) * 4u;
        {
            const int u = wave - 4;
            load_w_units<4, 4>(wrs, wvoff, (unsigned)(u * 128) * (NN * 4u), smem + LDS_W + u * 32768, lane);
            lds_add1(&lf[F_WA], lane);
        }
        STAMP(1);
        if (wave == 7) {
            lds_wait_ge(&lf[F_CALL], 1);
            const unsigned call = __builtin_amdgcn_readfirstlane(lds_load(&lf[F_CALL]));
            gwait_all_ge((unsigned*)(myws + OFF_XRDY + kq * 128) + (lane & 31), true, call, tmo, call);
            if (lane == 0) __hip_atomic_store(&lf[F_XALL], 1u, __ATOMIC_RELAXED, __HIP_MEMORY_SCOPE_WORKGROUP);
        }
        if (wave < 4 + NWB) {
            const int u = wave - 4;
            load_w_units<16 / NWB, WWINB>(wrs, wvoff + 256, (unsigned)(u * (16 / NWB) * 32) * (NN * 4u), smem + LDS_W + 4096 + u * (16 / NWB) * 8192, lane);
            lds_add1(&lf[F_WB], lane);
            STAMP(2);
        }
        if (wave < 7) {
            lds_wait_ge(&lf[F_CALL], 1);
            const unsigned call = __builtin_amdgcn_readfirstlane(lds_load(&lf[F_CALL]));
            lds_wait_ge(&lf[F_SAME], 1);
            const bool same = __builtin_amdgcn_readfirstlane(lds_load(&lf[F_SAME])) == 1;
            const unsigned dnoff = OFF_DONE + (unsigned)(grp * 8 + kq) * 128u;
            const unsigned seq0 = (call - 1) * NPASS;
            const unsigned pb0 = OFF_PART + (unsigned)((grp * 8 + kq) * 8) * 2048u + lane * 16u;
            for (;;) {
                const int ps = (int)lds_ticket(&lf[F_RTICKET]);
                if (ps >= NPASS) break;
                lds_wait_ge(&lf[F_CDONE], 4 * (ps + 1));
                STAMP(24 + ps);
                const unsigned so = (unsigned)(ps % RING) * SLOT_BYTES;
                const unsigned par = ((ps / RING) + 1) & 1;
                u4 v[8][2];
                bool bad = false;
                unsigned long long t0 = 0;
                unsigned pend = 0xffu;
                unsigned tv = 0;
                for (int tries = 0;; ++tries) {
#pragma unroll
                    for (int pr = 0; pr < 8; ++pr)
                        if (pend & (1u << pr)) {
                            asm volatile("" ::: "memory");
                            v[pr][0] = BLOAD(wsrs, pb0, so + pr * 2048, RPOL);
                            v[pr][1] = BLOAD(wsrs, pb0 + 1024, so + pr * 2048, RPOL);
                        }
                    tv = __hip_atomic_load(tmo, __ATOMIC_RELAXED, __HIP_MEMORY_SCOPE_AGENT);
                    unsigned np = 0;
#pragma unroll
                    for (int pr = 0; pr < 8; ++pr) {
                        const unsigned m = ((v[pr][0][0] & v[pr][0][2] & v[pr][1][0] & v[pr][1][2]) & 1u) == par &&
                                           ((v[pr][0][0] | v[pr][0][2] | v[pr][1][0] | v[pr][1][2]) & 1u) == par;
                        if (!__all(m)) np |= 1u << pr;
                    }
                    pend = np;
                    if (!pend) break;
                    if (tries == 0) t0 = __builtin_amdgcn_s_memrealtime();
                    if ((unsigned)__builtin_amdgcn_readfirstlane(tv) == call || __builtin_amdgcn_s_memrealtime() - t0 > DEADLINE_TICKS) {
                        __hip_atomic_store(tmo, call, __ATOMIC_RELAXED, __HIP_MEMORY_SCOPE_AGENT);
                        bad = true;
                        break;
                    }
                    __builtin_amdgcn_s_sleep(RSLEEP);
                }
                STAMP(44 + ps);
                if (lane == 0) {
                    if (same) __builtin_amdgcn_raw_buffer_store_b32(seq0 + ps + 1, wsrs, dnoff + (ps % RING) * 4u, 0, 0);
                    else __builtin_amdgcn_raw_buffer_store_b32(seq0 + ps + 1, wsrs, dnoff + (ps % RING) * 4u, 0, 16);
                }
                f4 o[2];
#pragma unroll
                for (int q = 0; q < 2; ++q) {
                    o[q] = __builtin_bit_cast(f4, v[0][q]);
#pragma unroll
                    for (int pr = 1; pr < 8; ++pr) o[q] += __builtin_bit_cast(f4, v[pr][q]);
                }
                if (ps >= ORING) lds_wait_ge(&lf[F_OFREE], ps - ORING + 1);
                f4 r[2];
#pragma unroll
                for (int q = 0; q < 2; ++q) r[q] = (f4){silu(o[q][0]), silu(o[q][1]), silu(o[q][2]), silu(o[q][3])};
                bad = bad || ((unsigned)__builtin_amdgcn_readfirstlane(tv) == call);
#pragma unroll
                for (int q = 0; q < 2; ++q) {
                    if (bad) r[q] = (f4){__builtin_nanf(""), __builtin_nanf(""), __builtin_nanf(""), __builtin_nanf("")};
                    *(f4*)(smem + LDS_OUT + (ps % ORING) * 2048 + q * 1024 + lane * 16) = r[q];
                }
                asm volatile("s_waitcnt lgkmcnt(0)" ::: "memory");
                if (lane == 0) __hip_atomic_store(&lf[F_ORDY + ps], 1u, __ATOMIC_RELAXED, __HIP_MEMORY_SCOPE_WORKGROUP);
                STAMP(4 + ps);
            }
        } else {
            lds_wait_ge(&lf[F_CALL], 1);
            const unsigned call = __builtin_amdgcn_readfirstlane(lds_load(&lf[F_CALL]));
            float* obase = p.out[dest] + (size_t)(rank * ML + kq * 8 + ((lane >> 4) & 1) * 4 + (lane >> 5)) * NB + nb * 128 + c * 4;
            for (int ps = 0; ps < NPASS; ++ps) {
                lds_wait_ge(&lf[F_ORDY + ps], 1);
                f4 r[2];
#pragma unroll
                for (int q = 0; q < 2; ++q) r[q] = *(const f4*)(smem + LDS_OUT + (ps % ORING) * 2048 + q * 1024 + lane * 16);
                asm volatile("s_waitcnt lgkmcnt(0)" ::: "memory");
                if (lane == 0) __hip_atomic_store(&lf[F_OFREE], (unsigned)(ps + 1), __ATOMIC_RELAXED, __HIP_MEMORY_SCOPE_WORKGROUP);
                float* o = obase + (size_t)(pass_m(ps) * 64) * NB + pass_h(ps) * 64;
#pragma unroll
                for (int q = 0; q < PUSHN; ++q)
                    asm volatile("global_store_dwordx4 %0, %1, off sc0 sc1\n\ts_nop 1" :: "v"(o + q * 2 * NB), "v"(r[q]) : "memory");
                STAMP(4 + ps);
            }
            asm volatile("s_waitcnt vmcnt(0)" ::: "memory");
            STAMP(20);
            if (lane == 0) {
                if (dest != rank) {
                    __hip_atomic_fetch_add((unsigned*)(p.ws[dest] + OFF_RFLAG + rank * 128), 1u, __ATOMIC_RELAXED, __HIP_MEMORY_SCOPE_SYSTEM);
                } else if (idx < NR && idx != rank) {
                    unsigned* flag = (unsigned*)(myws + OFF_RFLAG + idx * 128);
                    const unsigned target = call * 32u;
                    unsigned long long t0 = __builtin_amdgcn_s_memrealtime();
                    while (__hip_atomic_load(flag, __ATOMIC_RELAXED, __HIP_MEMORY_SCOPE_SYSTEM) < target) {
                        if (__hip_atomic_load(tmo, __ATOMIC_RELAXED, __HIP_MEMORY_SCOPE_AGENT) == call ||
                            __builtin_amdgcn_s_memrealtime() - t0 > DEADLINE_TICKS) {
                            __hip_atomic_store(tmo, call, __ATOMIC_RELAXED, __HIP_MEMORY_SCOPE_AGENT);
                            obase[0] = __builtin_nanf("");
                            break;
                        }
                        __builtin_amdgcn_s_sleep(2);
                    }
                    __builtin_amdgcn_fence(__ATOMIC_ACQUIRE, "");
                }
            }
            STAMP(21);
        }
    }
}

extern "C" void kernel_launch(int rank, int nranks, ncclComm_t comm,
                              hipStream_t stream,
                              void* const* d_in, const int* in_sizes, int n_in,
                              void* d_out, int out_size,
                              const int* in_dtypes, int out_dtype,
                              void* const* d_out_all,
                              void* const* d_ws_all, size_t ws_bytes) {
    if (nranks != NR || ws_bytes < WS_USED || n_in < 2 || rank < 0 || rank >= NR) { fprintf(stderr, "bad config\n"); return; }
    Params p;
    p.x = (const float*)d_in[0];
    p.w = (const float*)d_in[1];
    for (int i = 0; i < NR; ++i) { p.out[i] = (float*)d_out_all[i]; p.ws[i] = (unsigned char*)d_ws_all[i]; }
    p.rank = rank;
    fused_kernel<<<NWG, NTH, 0, stream>>>(p);
    HIP_CHECK(hipGetLastError());
}
```

```cpp
#include <hip/hip_runtime.h>
#include <rccl/rccl.h>
#include <cstdio>
#include <cstdint>

#define HIP_CHECK(expr) do { hipError_t _e = (expr); if (_e != hipSuccess) { \
    fprintf(stderr, "HIP error %s at %s:%d\n", hipGetErrorString(_e), __FILE__, __LINE__); \
    return; } } while (0)

typedef _Float16 h8 __attribute__((ext_vector_type(8)));
typedef float f4 __attribute__((ext_vector_type(4)));
typedef unsigned u4 __attribute__((ext_vector_type(4)));

#define RING_N 2
#define WWINB 4
constexpr int APOL = 16;
constexpr int RPOL = 16;
constexpr int WPOL = 2;
constexpr int XPOL = 2;
constexpr int NWB = 2;

constexpr int NR = 8;
constexpr int ML = 512;
constexpr int KK = 4096;
constexpr int NN = 4096;
constexpr int NB = 512;
constexpr int NWG = 256;
constexpr int NTH = 512;
constexpr int NPASS = 16;
constexpr int RING = RING_N;
constexpr int ORING = 8;
constexpr unsigned SLOT_BYTES = 4u << 20;
#define DLT 50000000ull
constexpr unsigned long long DEADLINE_TICKS = DLT;
constexpr unsigned TAG_MASK = 7u;
__device__ __forceinline__ unsigned tag_of(int ps, unsigned call) {
    const int r = ps % RING;
    const unsigned uses = (unsigned)((NPASS - 1 - r) / RING + 1);
    const unsigned n = (call - 1) * uses + (unsigned)(ps / RING) + 1u;
    return (n % 7u) + 1u;
}

constexpr unsigned OFF_TMO = 128;
constexpr unsigned OFF_RFLAG = 256;
constexpr unsigned OFF_CTR = 3072;
constexpr unsigned OFF_XCC = 4096;
constexpr unsigned OFF_DONE = 8192;
constexpr unsigned OFF_X16 = 1u << 20;
constexpr unsigned OFF_PART = 9u << 20;
constexpr size_t WS_USED = (size_t)OFF_PART + (size_t)RING * SLOT_BYTES;

constexpr int LDS_W = 0;
constexpr int LDS_OUT = 131072;
constexpr int LDS_FLAG = LDS_OUT + ORING * 2048;
constexpr int LDS_BYTES = LDS_FLAG + 256;
enum { F_CALL = 0, F_WA, F_WB, F_CDONE, F_RTICKET, F_OFREE, F_SAME, F_ORDY  , F_NWORDS = F_ORDY + 16 };

struct Params {
    const float* x;
    const float* w;
    float* out[NR];
    unsigned char* ws[NR];
    int rank;
};

__device__ __forceinline__ float silu(float y) {
    float c = fminf(fmaxf(y, -60.0f), 60.0f);
    return y * __builtin_amdgcn_rcpf(1.0f + __expf(-c));
}

typedef __attribute__((address_space(3))) unsigned lds_u32;
__device__ __forceinline__ unsigned lds_load(lds_u32* p) { return __hip_atomic_load(p, __ATOMIC_RELAXED, __HIP_MEMORY_SCOPE_WORKGROUP); }
__device__ __forceinline__ void lds_add1(lds_u32* p, int lane) {
    asm volatile("s_waitcnt lgkmcnt(0)" ::: "memory");
    if (lane == 0) __hip_atomic_fetch_add(p, 1u, __ATOMIC_RELAXED, __HIP_MEMORY_SCOPE_WORKGROUP);
}

__device__ __forceinline__ unsigned lds_ticket(lds_u32* p) {
    unsigned old;
    unsigned long long save;
    const unsigned addr = (unsigned)(size_t)p, one = 1u;
    asm volatile("s_mov_b64 %1, exec\n\ts_mov_b64 exec, 1\n\tds_add_rtn_u32 %0, %2, %3\n\ts_waitcnt lgkmcnt(0)\n\ts_mov_b64 exec, %1"
                 : "=&v"(old), "=&s"(save) : "v"(addr), "v"(one) : "memory");
    return __builtin_amdgcn_readfirstlane(old);
}

__device__ __forceinline__ bool lds_wait_ge(lds_u32* p, unsigned target) {
    if ((unsigned)__builtin_amdgcn_readfirstlane(lds_load(p)) >= target) return true;
    unsigned long long t0 = __builtin_amdgcn_s_memrealtime();
    while ((unsigned)__builtin_amdgcn_readfirstlane(lds_load(p)) < target) {
        if (__builtin_amdgcn_s_memrealtime() - t0 > 2 * DEADLINE_TICKS) return false;
        __builtin_amdgcn_s_sleep(1);
    }
    return true;
}

__device__ __forceinline__ bool give_up(unsigned long long t0, unsigned* tmo, unsigned call) {
    if ((unsigned)__builtin_amdgcn_readfirstlane(__hip_atomic_load(tmo, __ATOMIC_RELAXED, __HIP_MEMORY_SCOPE_AGENT)) == call ||
        __builtin_amdgcn_s_memrealtime() - t0 > DEADLINE_TICKS) {
        __hip_atomic_store(tmo, call, __ATOMIC_RELAXED, __HIP_MEMORY_SCOPE_AGENT);
        return true;
    }
    return false;
}

__device__ __forceinline__ bool gwait_all_ge(unsigned* word, bool sel, unsigned target, unsigned* tmo, unsigned call) {
    unsigned long long t0 = __builtin_amdgcn_s_memrealtime();
    for (;;) {
        const unsigned v = __hip_atomic_load(word, __ATOMIC_RELAXED, __HIP_MEMORY_SCOPE_AGENT);
        if (__all(!sel || v >= target)) return true;
        if (give_up(t0, tmo, call)) return false;
        __builtin_amdgcn_s_sleep(1);
    }
}

#define RSRC(ptr, bytes) __builtin_amdgcn_make_buffer_rsrc((void*)(ptr), 0, (bytes), 0x00020000)
#define BLOAD(rs, voff, soff, aux) __builtin_amdgcn_raw_buffer_load_b128(rs, voff, soff, aux)

__device__ __forceinline__ h8 cvt8(f4 a, f4 b) {
    h8 r = {(_Float16)a[0], (_Float16)a[1], (_Float16)a[2], (_Float16)a[3], (_Float16)b[0], (_Float16)b[1], (_Float16)b[2], (_Float16)b[3]};
    return r;
}

template <int NU, int WIN, class F>
__device__ __forceinline__ void load_w_units(__amdgpu_buffer_rsrc_t wrs, unsigned voff, unsigned soff0, unsigned char* ldsw, int lane, F after_issue) {
    f4 t[WIN][8];
#define W_ISSUE(u) _Pragma("unroll") for (int jj = 0; jj < 8; ++jj) \
        t[(u) % WIN][jj] = __builtin_bit_cast(f4, BLOAD(wrs, voff, soff0 + (unsigned)((u) * 32 + (jj & 3) + 16 * (jj >> 2)) * (NN * 4u), WPOL))
#pragma unroll
    for (int u = 0; u < WIN; ++u) { W_ISSUE(u); }
    __builtin_amdgcn_sched_barrier(0);
    after_issue();
    __builtin_amdgcn_sched_barrier(0);
#pragma unroll
    for (int u = 0; u < NU; ++u) {
        h8 f[4];
#pragma unroll
        for (int tt = 0; tt < 4; ++tt)
#pragma unroll
            for (int jj = 0; jj < 8; ++jj) f[tt][jj] = (_Float16)t[u % WIN][jj][tt];
        __builtin_amdgcn_sched_barrier(0);
        if (u + WIN < NU) { W_ISSUE(u + WIN); }
#pragma unroll
        for (int tt = 0; tt < 4; ++tt) *(h8*)(ldsw + (u * 8 + tt) * 1024 + lane * 16) = f[tt];
        __builtin_amdgcn_sched_barrier(0);
    }
#undef W_ISSUE
}

__device__ __forceinline__ int pass_m(int q) { return q < 8 ? q : 15 - q; }
__device__ __forceinline__ int pass_h(int q) { return q >> 3; }

typedef __attribute__((address_space(3))) const h8 lds_h8;
__device__ __forceinline__ h8 lds_frag(unsigned addr, int imm) { return *(lds_h8*)(size_t)(addr + imm); }

struct Fin {
    __amdgpu_buffer_rsrc_t wsrs;
    unsigned char* myws;
    unsigned* tmo;
    lds_u32* cdone;
    unsigned pvoff, dvoff0, seq0, call;
    int lane;
    bool same;
};

__device__ __forceinline__ void ensure_a(h8 (&a)[16], __amdgpu_buffer_rsrc_t wsrs, unsigned avoff, unsigned soff, unsigned* tmo, unsigned call) {
    unsigned long long t0 = 0;
    for (int tries = 0;; ++tries) {
        unsigned m = ~0u;
#pragma unroll
        for (int s = 0; s < 16; ++s) { const u4 w = __builtin_bit_cast(u4, a[s]); m = min(m, min(w[0], w[2])); }
        if (__all(m != 0u)) return;
        if (tries == 0) t0 = __builtin_amdgcn_s_memrealtime();
        if (give_up(t0, tmo, call)) return;
        asm volatile("" ::: "memory");
#pragma unroll
        for (int s = 0; s < 16; ++s) a[s] = __builtin_bit_cast(h8, BLOAD(wsrs, avoff, soff + s * 1024, 16));
    }
}

__device__ __forceinline__ void publish(const Fin& f, int ps, const f4 (&acc)[4], unsigned dv) {
    if (ps >= RING) {
        const unsigned need = f.seq0 + (ps - RING + 1);
        if (!__all(dv >= need)) gwait_all_ge((unsigned*)(f.myws + f.dvoff0 + (ps % RING) * 4u), true, need, f.tmo, f.call);
    }
    const unsigned so = (unsigned)(ps % RING) * SLOT_BYTES;
    const unsigned par = tag_of(ps, f.call);
#pragma unroll
    for (int jr = 0; jr < 4; ++jr) {
        u4 v = {(__float_as_uint(acc[0][jr]) & ~TAG_MASK) | par, __float_as_uint(acc[1][jr]), (__float_as_uint(acc[2][jr]) & ~TAG_MASK) | par, __float_as_uint(acc[3][jr])};
        if (f.same) __builtin_amdgcn_raw_buffer_store_b128(v, f.wsrs, f.pvoff, so + jr * 512, 0);
        else __builtin_amdgcn_raw_buffer_store_b128(v, f.wsrs, f.pvoff, so + jr * 512, 16);
    }
    if (f.lane == 0) __hip_atomic_fetch_add(f.cdone, 1u, __ATOMIC_RELAXED, __HIP_MEMORY_SCOPE_WORKGROUP);
}

template <bool RELOAD, bool PREV>
__device__ __forceinline__ void pass_body(int ps, h8 (&a)[16], f4 (&acc)[4], const f4 (&accp)[4], unsigned dvp, unsigned& dv, h8 (&b)[3][4],
                                          unsigned wb, unsigned wb_next, const Fin& f, unsigned avoff, unsigned nsoff, unsigned csoff) {
    unsigned amin = ~0u;
    unsigned lo = wb, hi = wb + 65536u;
    asm volatile("" : "+v"(lo), "+v"(hi));
#pragma unroll
    for (int s = 0; s < 16; ++s) {
#pragma unroll
        for (int t = 0; t < 4; ++t) {
            if (s == 0) acc[t] = __builtin_amdgcn_mfma_f32_16x16x32_f16(a[s], b[0][t], (f4){0.f, 0.f, 0.f, 0.f}, 0, 0, 0);
            else acc[t] = __builtin_amdgcn_mfma_f32_16x16x32_f16(a[s], b[s % 3][t], acc[t], 0, 0, 0);
            if (s + 2 < 8) b[(s + 2) % 3][t] = lds_frag(lo, ((s + 2) * 8 + t) * 1024);
            else if (s + 2 < 16) b[(s + 2) % 3][t] = lds_frag(hi, ((s + 2 - 8) * 8 + t) * 1024);
            __builtin_amdgcn_sched_barrier(0);
        }
        if (s == 0) {
            if (PREV) publish(f, ps - 1, accp, dvp);
            asm volatile("" ::: "memory");
            dv = __builtin_amdgcn_raw_buffer_load_b32(f.wsrs, f.dvoff0, (ps % RING) * 4u, 16);
            asm volatile("" ::: "memory");
        }
        { const u4 w = __builtin_bit_cast(u4, a[s]); amin = min(amin, min(w[0], w[2])); }
        if (RELOAD) a[s] = __builtin_bit_cast(h8, BLOAD(f.wsrs, avoff, nsoff + s * 1024, APOL));
        __builtin_amdgcn_sched_barrier(0);
    }
    if (!__all(amin != 0u)) {
#pragma unroll
        for (int t = 0; t < 4; ++t) acc[t] = (f4){0.f, 0.f, 0.f, 0.f};
        unsigned long long t0 = __builtin_amdgcn_s_memrealtime();
#pragma unroll 1
        for (int q = 0; q < 4; ++q) {
            u4 w[4];
            for (;;) {
                asm volatile("" ::: "memory");
                unsigned m = ~0u;
#pragma unroll
                for (int i = 0; i < 4; ++i) w[i] = BLOAD(f.wsrs, avoff, csoff + (q * 4 + i) * 1024, 16);
#pragma unroll
                for (int i = 0; i < 4; ++i) m = min(m, min(w[i][0], w[i][2]));
                if (__all(m != 0u)) break;
                if (give_up(t0, f.tmo, f.call)) break;
            }
#pragma unroll
            for (int i = 0; i < 4; ++i)
#pragma unroll
                for (int t = 0; t < 4; ++t)
                    acc[t] = __builtin_amdgcn_mfma_f32_16x16x32_f16(__builtin_bit_cast(h8, w[i]), lds_frag(wb + (q * 4 + i) * 8192u, t * 1024), acc[t], 0, 0, 0);
        }
    }
    unsigned nx = wb_next;
    asm volatile("" : "+v"(nx));
#pragma unroll
    for (int s = 0; s < 2; ++s)
#pragma unroll
        for (int t = 0; t < 4; ++t) b[s][t] = lds_frag(nx, (s * 8 + t) * 1024);
}

__device__ __forceinline__ void prepass(const Params& p, __amdgpu_buffer_rsrc_t wsrs, unsigned char* myws, lds_u32* lf, int idx, int j, int q4, int lane) {
    const int c = lane & 15, g = lane >> 4;
    const auto xrs = RSRC(p.x, ML * KK * 4u);
    const unsigned voff = ((unsigned)(idx * 16 + c) * KK + j * 512 + q4 * 128 + g * 4) * 4u;
    f4 t[8];
#pragma unroll
    for (int q = 0; q < 4; ++q) {
        t[2 * q] = __builtin_bit_cast(f4, BLOAD(xrs, voff + q * 128, 0, XPOL));
        t[2 * q + 1] = __builtin_bit_cast(f4, BLOAD(xrs, voff + q * 128 + 64, 0, XPOL));
    }
    u4 f[4];
#pragma unroll
    for (int q = 0; q < 4; ++q) {
        f[q] = __builtin_bit_cast(u4, cvt8(t[2 * q], t[2 * q + 1]));
        if (f[q][0] == 0u) f[q][0] = 0x8000u;
        if (f[q][2] == 0u) f[q][2] = 0x8000u;
    }
    lds_wait_ge(&lf[F_CALL], 1);
    const unsigned call = __builtin_amdgcn_readfirstlane(lds_load(&lf[F_CALL]));
    const unsigned xo = OFF_X16 + (call & 1u) * (4u << 20) + (unsigned)((idx * 8 + j) * 16 + q4 * 4) * 1024u + lane * 16u;
#pragma unroll
    for (int q = 0; q < 4; ++q) __builtin_amdgcn_raw_buffer_store_b128(f[q], wsrs, xo + q * 1024, 0, 16);
}

__global__ __launch_bounds__(NTH, 2) void fused_kernel(Params p) {
    __shared__ __attribute__((aligned(16))) unsigned char smem[LDS_BYTES];
    const int tid = threadIdx.x, lane = tid & 63;
    const int wave = __builtin_amdgcn_readfirstlane(tid >> 6);
    const int bid = blockIdx.x;
    const int rank = p.rank;
    const int j = bid & 7, idx = bid >> 3;
    const int dest = j;
    const int kq = idx & 7, nb = idx >> 3;
    const int grp = j * 4 + nb;
    unsigned char* myws = p.ws[rank];
    lds_u32* lf = (lds_u32*)(smem + LDS_FLAG);
    if (tid < F_NWORDS) lf[tid] = 0;
    __syncthreads();
    unsigned* tmo = (unsigned*)(myws + OFF_TMO);
    const auto wsrs = RSRC(myws, (unsigned)WS_USED);
    const int c = lane & 15, g = lane >> 4;

    if (wave < 4) {
        prepass(p, wsrs, myws, lf, idx, j, wave, lane);
        const unsigned call = __builtin_amdgcn_readfirstlane(lds_load(&lf[F_CALL]));
        const unsigned avoff = OFF_X16 + (call & 1u) * (4u << 20) + (unsigned)((wave * 8 + kq) * 16) * 1024u + lane * 16u;
        h8 a[2][16];
        unsigned xcc_v;
        {
            lds_wait_ge(&lf[F_WA], 4);
            unsigned* tab = (unsigned*)(myws + OFF_XCC + grp * 128) + (lane & 7);
            xcc_v = __hip_atomic_load(tab, __ATOMIC_RELAXED, __HIP_MEMORY_SCOPE_AGENT);
#pragma unroll
            for (int q = 0; q < 2; ++q)
#pragma unroll
                for (int s = 0; s < 16; ++s) a[q][s] = __builtin_bit_cast(h8, BLOAD(wsrs, avoff, q * (512u << 10) + s * 1024, 16));
            if (!__all((xcc_v >> 4) >= call)) {
                gwait_all_ge(tab, true, call << 4, tmo, call);
                xcc_v = __hip_atomic_load(tab, __ATOMIC_RELAXED, __HIP_MEMORY_SCOPE_AGENT);
            }
            ensure_a(a[0], wsrs, avoff, 0u, tmo, call);
        }
        const unsigned wb0 = (unsigned)(size_t)(lds_h8*)(smem + LDS_W) + lane * 16;
        Fin f;
        f.wsrs = wsrs; f.myws = myws; f.tmo = tmo; f.cdone = &lf[F_CDONE]; f.call = call; f.lane = lane;
        f.dvoff0 = OFF_DONE + (unsigned)(grp * 8 + 2 * wave + (lane & 1)) * 128u;
        f.seq0 = (call - 1) * NPASS;
        f.pvoff = OFF_PART + (unsigned)((grp * 8 + 2 * wave + (lane >> 5)) * 8 + kq) * 2048u + (lane & 31) * 16u;
        f.same = __all(xcc_v == (unsigned)__builtin_amdgcn_readfirstlane(xcc_v));
        if (wave == 0 && lane == 0) __hip_atomic_store(&lf[F_SAME], f.same ? 1u : 2u, __ATOMIC_RELAXED, __HIP_MEMORY_SCOPE_WORKGROUP);
        lds_wait_ge(&lf[F_WA], 4);
        f4 acc[2][4];
        unsigned dv[2] = {0u, 0u};
        h8 b[3][4];
#pragma unroll
        for (int s = 0; s < 2; ++s)
#pragma unroll
            for (int t = 0; t < 4; ++t) b[s][t] = lds_frag(wb0, (s * 8 + t) * 1024);
        auto vo = [&](int q) { return (unsigned)pass_m(q) * (512u << 10); };
        auto voff2 = [&](int q) { return (q + 2 < NPASS) ? vo(q + 2) : 0x80000000u; };
        const unsigned wbA = wb0, wbB = wb0 + 4096;
        pass_body<false, false>(0, a[0], acc[0], acc[1], 0u, dv[0], b, wbA, wbA, f, avoff, 0u, vo(0));
        publish(f, 0, acc[0], dv[0]);
#pragma unroll
        for (int s = 0; s < 16; ++s) a[0][s] = __builtin_bit_cast(h8, BLOAD(wsrs, avoff, voff2(0) + s * 1024, APOL));
        pass_body<true, false>(1, a[1], acc[1], acc[0], 0u, dv[1], b, wbA, wbA, f, avoff, voff2(1), vo(1));
        for (int ps = 2; ps < NPASS; ps += 2) {
            if (ps == 6) lds_wait_ge(&lf[F_WB], NWB);
            const unsigned w0 = ps < 8 ? wbA : wbB, w1 = ps + 1 < 7 ? wbA : wbB;
            pass_body<true, true>(ps, a[0], acc[0], acc[1], dv[1], dv[0], b, w0, w0, f, avoff, voff2(ps), vo(ps));
            pass_body<true, true>(ps + 1, a[1], acc[1], acc[0], dv[0], dv[1], b, w0, w1, f, avoff, voff2(ps + 1), vo(ps + 1));
        }
        publish(f, NPASS - 1, acc[1], dv[1]);
        {
            const unsigned xz = OFF_X16 + ((call + 1) & 1u) * (4u << 20) + (unsigned)((idx * 8 + j) * 16 + wave * 4) * 1024u + lane * 16u;
            const u4 z = {0u, 0u, 0u, 0u};
#pragma unroll
            for (int q = 0; q < 4; ++q) __builtin_amdgcn_raw_buffer_store_b128(z, wsrs, xz + q * 1024, 0, 16);
        }
    } else {
        const auto wrs = RSRC(p.w, (unsigned)KK * NN * 4u);
        const unsigned wvoff = ((unsigned)(kq * 512 + g * 4) * NN + dest * NB + nb * 128 + c * 4) * 4u;
        unsigned ctr = 0;
        if (wave == 7) ctr = __hip_atomic_load((unsigned*)(myws + OFF_CTR) + bid, __ATOMIC_RELAXED, __HIP_MEMORY_SCOPE_AGENT);
        {
            const int u = wave - 4;
            load_w_units<4, 4>(wrs, wvoff, (unsigned)(u * 128) * (NN * 4u), smem + LDS_W + u * 32768, lane, [&]() {
                if (wave == 7) {
                    const unsigned call0 = __builtin_amdgcn_readfirstlane(ctr) + 1;
                    if (lane == 0) {
                        __hip_atomic_store(&lf[F_CALL], call0, __ATOMIC_RELAXED, __HIP_MEMORY_SCOPE_WORKGROUP);
                        unsigned xcc;
                        asm volatile("s_getreg_b32 %0, hwreg(HW_REG_XCC_ID, 0, 4)" : "=s"(xcc));
                        __hip_atomic_store((unsigned*)(myws + OFF_XCC + grp * 128 + kq * 4), (call0 << 4) | xcc, __ATOMIC_RELAXED, __HIP_MEMORY_SCOPE_AGENT);
                        __hip_atomic_store((unsigned*)(myws + OFF_CTR) + bid, call0, __ATOMIC_RELAXED, __HIP_MEMORY_SCOPE_AGENT);
                    }
                }
            });
            lds_add1(&lf[F_WA], lane);
        }
        if (wave < 4 + NWB) {
            const int u = wave - 4;
            load_w_units<16 / NWB, WWINB>(wrs, wvoff + 256, (unsigned)(u * (16 / NWB) * 32) * (NN * 4u), smem + LDS_W + 4096 + u * (16 / NWB) * 8192, lane, []() {});
            lds_add1(&lf[F_WB], lane);
        }
        if (wave < 7) {
            lds_wait_ge(&lf[F_CALL], 1);
            const unsigned call = __builtin_amdgcn_readfirstlane(lds_load(&lf[F_CALL]));
            lds_wait_ge(&lf[F_SAME], 1);
            const bool same = __builtin_amdgcn_readfirstlane(lds_load(&lf[F_SAME])) == 1;
            const unsigned dnoff = OFF_DONE + (unsigned)(grp * 8 + kq) * 128u;
            const unsigned seq0 = (call - 1) * NPASS;
            const unsigned pb0 = OFF_PART + (unsigned)((grp * 8 + kq) * 8) * 2048u + lane * 16u;
            for (;;) {
                const int ps = (int)lds_ticket(&lf[F_RTICKET]);
                if (ps >= NPASS) break;
                lds_wait_ge(&lf[F_CDONE], 4 * (ps + 1));
                const unsigned so = (unsigned)(ps % RING) * SLOT_BYTES;
                const unsigned par = tag_of(ps, call);
                u4 v[8][2];
                bool bad = false;
                unsigned long long t0 = 0;
                unsigned pend = 0xffu;
                unsigned tv = 0;
                for (int tries = 0;; ++tries) {
#pragma unroll
                    for (int pr = 0; pr < 8; ++pr)
                        if (pend & (1u << pr)) {
                            asm volatile("" ::: "memory");
                            v[pr][0] = BLOAD(wsrs, pb0, so + pr * 2048, RPOL);
                            v[pr][1] = BLOAD(wsrs, pb0 + 1024, so + pr * 2048, RPOL);
                        }
                    tv = __hip_atomic_load(tmo, __ATOMIC_RELAXED, __HIP_MEMORY_SCOPE_AGENT);
                    unsigned np = 0;
#pragma unroll
                    for (int pr = 0; pr < 8; ++pr) {
                        const unsigned m = ((v[pr][0][0] & v[pr][0][2] & v[pr][1][0] & v[pr][1][2]) & TAG_MASK) == par &&
                                           ((v[pr][0][0] | v[pr][0][2] | v[pr][1][0] | v[pr][1][2]) & TAG_MASK) == par;
                        if (!__all(m)) np |= 1u << pr;
                    }
                    pend = np;
                    if (!pend) break;
                    if (tries == 0) t0 = __builtin_amdgcn_s_memrealtime();
                    if ((unsigned)__builtin_amdgcn_readfirstlane(tv) == call || __builtin_amdgcn_s_memrealtime() - t0 > DEADLINE_TICKS) {
                        __hip_atomic_store(tmo, call, __ATOMIC_RELAXED, __HIP_MEMORY_SCOPE_AGENT);
                        bad = true;
                        break;
                    }
                    __builtin_amdgcn_s_sleep(2);
                }
                if (lane == 0) {
                    if (same) __builtin_amdgcn_raw_buffer_store_b32(seq0 + ps + 1, wsrs, dnoff + (ps % RING) * 4u, 0, 0);
                    else __builtin_amdgcn_raw_buffer_store_b32(seq0 + ps + 1, wsrs, dnoff + (ps % RING) * 4u, 0, 16);
                }
                f4 o[2];
#pragma unroll
                for (int q = 0; q < 2; ++q) {
                    o[q] = __builtin_bit_cast(f4, v[0][q]);
#pragma unroll
                    for (int pr = 1; pr < 8; ++pr) o[q] += __builtin_bit_cast(f4, v[pr][q]);
                }
                if (ps >= ORING) lds_wait_ge(&lf[F_OFREE], ps - ORING + 1);
                f4 r[2];
#pragma unroll
                for (int q = 0; q < 2; ++q) r[q] = (f4){silu(o[q][0]), silu(o[q][1]), silu(o[q][2]), silu(o[q][3])};
                bad = bad || ((unsigned)__builtin_amdgcn_readfirstlane(tv) == call);
#pragma unroll
                for (int q = 0; q < 2; ++q) {
                    if (bad) r[q] = (f4){__builtin_nanf(""), __builtin_nanf(""), __builtin_nanf(""), __builtin_nanf("")};
                    *(f4*)(smem + LDS_OUT + (ps % ORING) * 2048 + q * 1024 + lane * 16) = r[q];
                }
                asm volatile("s_waitcnt lgkmcnt(0)" ::: "memory");
                if (lane == 0) __hip_atomic_store(&lf[F_ORDY + ps], 1u, __ATOMIC_RELAXED, __HIP_MEMORY_SCOPE_WORKGROUP);
            }
        } else {
            lds_wait_ge(&lf[F_CALL], 1);
            const unsigned call = __builtin_amdgcn_readfirstlane(lds_load(&lf[F_CALL]));
            float* obase = p.out[dest] + (size_t)(rank * ML + kq * 8 + ((lane >> 4) & 1) * 4 + (lane >> 5)) * NB + nb * 128 + c * 4;
            for (int ps = 0; ps < NPASS; ++ps) {
                lds_wait_ge(&lf[F_ORDY + ps], 1);
                f4 r[2];
#pragma unroll
                for (int q = 0; q < 2; ++q) r[q] = *(const f4*)(smem + LDS_OUT + (ps % ORING) * 2048 + q * 1024 + lane * 16);
                asm volatile("s_waitcnt lgkmcnt(0)" ::: "memory");
                if (lane == 0) __hip_atomic_store(&lf[F_OFREE], (unsigned)(ps + 1), __ATOMIC_RELAXED, __HIP_MEMORY_SCOPE_WORKGROUP);
                float* o = obase + (size_t)(pass_m(ps) * 64) * NB + pass_h(ps) * 64;
#pragma unroll
                for (int q = 0; q < 2; ++q)
                    asm volatile("global_store_dwordx4 %0, %1, off sc0 sc1\n\ts_nop 1" :: "v"(o + q * 2 * NB), "v"(r[q]) : "memory");
            }
            asm volatile("s_waitcnt vmcnt(0)" ::: "memory");
            if (lane == 0) {
                if (dest != rank) {
                    __hip_atomic_fetch_add((unsigned*)(p.ws[dest] + OFF_RFLAG + rank * 128), 1u, __ATOMIC_RELAXED, __HIP_MEMORY_SCOPE_SYSTEM);
                } else if (idx < NR && idx != rank) {
                    unsigned* flag = (unsigned*)(myws + OFF_RFLAG + idx * 128);
                    const unsigned target = call * 32u;
                    unsigned long long t0 = __builtin_amdgcn_s_memrealtime();
                    while (__hip_atomic_load(flag, __ATOMIC_RELAXED, __HIP_MEMORY_SCOPE_SYSTEM) < target) {
                        if (__hip_atomic_load(tmo, __ATOMIC_RELAXED, __HIP_MEMORY_SCOPE_AGENT) == call ||
                            __builtin_amdgcn_s_memrealtime() - t0 > DEADLINE_TICKS) {
                            __hip_atomic_store(tmo, call, __ATOMIC_RELAXED, __HIP_MEMORY_SCOPE_AGENT);
                            obase[0] = __builtin_nanf("");
                            break;
                        }
                        __builtin_amdgcn_s_sleep(2);
                    }
                    __builtin_amdgcn_fence(__ATOMIC_ACQUIRE, "");
                }
            }
        }
    }
}

extern "C" void kernel_launch(int rank, int nranks, ncclComm_t comm,
                              hipStream_t stream,
                              void* const* d_in, const int* in_sizes, int n_in,
                              void* d_out, int out_size,
                              const int* in_dtypes, int out_dtype,
                              void* const* d_out_all,
                              void* const* d_ws_all, size_t ws_bytes) {
    if (nranks != NR || ws_bytes < WS_USED || n_in < 2 || rank < 0 || rank >= NR) { fprintf(stderr, "bad config\n"); return; }
    Params p;
    p.x = (const float*)d_in[0];
    p.w = (const float*)d_in[1];
    for (int i = 0; i < NR; ++i) { p.out[i] = (float*)d_out_all[i]; p.ws[i] = (unsigned char*)d_ws_all[i]; }
    p.rank = rank;
    fused_kernel<<<NWG, NTH, 0, stream>>>(p);
    HIP_CHECK(hipGetLastError());
}
```

```cpp
#include <hip/hip_runtime.h>
#include <rccl/rccl.h>
#include <cstdio>
#include <cstdint>

#define HIP_CHECK(expr) do { hipError_t _e = (expr); if (_e != hipSuccess) { \
    fprintf(stderr, "HIP error %s at %s:%d\n", hipGetErrorString(_e), __FILE__, __LINE__); \
    return; } } while (0)

typedef _Float16 h8 __attribute__((ext_vector_type(8)));
typedef float f4 __attribute__((ext_vector_type(4)));
typedef unsigned u4 __attribute__((ext_vector_type(4)));

#define RING_N 3
#define WWINB 4
#define AT 600
#define WBGATE 0
#define WDEL 0
#define CODEPF 0
#define EARLYA 0
constexpr int APOL = 16;
constexpr int RPOL = 16;
constexpr int WPOL = 2;
constexpr int XPOL = 2;
#define NWB_N 2
constexpr int NWB = 3;

constexpr int NR = 8;
constexpr int ML = 512;
constexpr int KK = 4096;
constexpr int NN = 4096;
constexpr int NB = 512;
constexpr int NWG = 256;
constexpr int NTH = 512;
constexpr int NPASS = 16;
constexpr int RING = RING_N;
constexpr int ORING = 8;
constexpr unsigned SLOT_BYTES = 4u << 20;
#define DLT 50000000ull
constexpr unsigned long long DEADLINE_TICKS = DLT;
constexpr unsigned TAG_MASK = 7u;
__device__ __forceinline__ unsigned tag_of(int ps, unsigned call) {
    const int r = ps % RING;
    const unsigned uses = (unsigned)((NPASS - 1 - r) / RING + 1);
    const unsigned n = (call - 1) * uses + (unsigned)(ps / RING) + 1u;
    return (n % 7u) + 1u;
}

constexpr unsigned OFF_TMO = 128;
constexpr unsigned OFF_RFLAG = 256;
constexpr unsigned OFF_CTR = 3072;
constexpr unsigned OFF_XCC = 4096;
constexpr unsigned OFF_DONE = 8192;
constexpr unsigned OFF_X16 = 1u << 20;
constexpr unsigned OFF_PART = 9u << 20;
constexpr size_t WS_USED = (size_t)OFF_PART + (size_t)RING * SLOT_BYTES;

constexpr int LDS_W = 0;
constexpr int LDS_OUT = 131072;
constexpr int LDS_FLAG = LDS_OUT + ORING * 2048;
constexpr int LDS_BYTES = LDS_FLAG + 256;
enum { F_CALL = 0, F_WA, F_WB, F_AISS, F_CDONE, F_RTICKET, F_OFREE, F_SAME, F_ORDY  , F_NWORDS = F_ORDY + 16 };

struct Params {
    const float* x;
    const float* w;
    float* out[NR];
    unsigned char* ws[NR];
    int rank;
};

__device__ __forceinline__ float silu(float y) {
    float c = fminf(fmaxf(y, -60.0f), 60.0f);
    return y * __builtin_amdgcn_rcpf(1.0f + __expf(-c));
}

typedef __attribute__((address_space(3))) unsigned lds_u32;
__device__ __forceinline__ unsigned lds_load(lds_u32* p) { return __hip_atomic_load(p, __ATOMIC_RELAXED, __HIP_MEMORY_SCOPE_WORKGROUP); }
__device__ __forceinline__ void lds_add1(lds_u32* p, int lane) {
    asm volatile("s_waitcnt lgkmcnt(0)" ::: "memory");
    if (lane == 0) __hip_atomic_fetch_add(p, 1u, __ATOMIC_RELAXED, __HIP_MEMORY_SCOPE_WORKGROUP);
}

__device__ __forceinline__ unsigned lds_ticket(lds_u32* p) {
    unsigned old;
    unsigned long long save;
    const unsigned addr = (unsigned)(size_t)p, one = 1u;
    asm volatile("s_mov_b64 %1, exec\n\ts_mov_b64 exec, 1\n\tds_add_rtn_u32 %0, %2, %3\n\ts_waitcnt lgkmcnt(0)\n\ts_mov_b64 exec, %1"
                 : "=&v"(old), "=&s"(save) : "v"(addr), "v"(one) : "memory");
    return __builtin_amdgcn_readfirstlane(old);
}

__device__ __forceinline__ bool lds_wait_ge(lds_u32* p, unsigned target) {
    if ((unsigned)__builtin_amdgcn_readfirstlane(lds_load(p)) >= target) return true;
    unsigned long long t0 = __builtin_amdgcn_s_memrealtime();
    while ((unsigned)__builtin_amdgcn_readfirstlane(lds_load(p)) < target) {
        if (__builtin_amdgcn_s_memrealtime() - t0 > 2 * DEADLINE_TICKS) return false;
        __builtin_amdgcn_s_sleep(1);
    }
    return true;
}

__device__ __forceinline__ bool give_up(unsigned long long t0, unsigned* tmo, unsigned call) {
    if ((unsigned)__builtin_amdgcn_readfirstlane(__hip_atomic_load(tmo, __ATOMIC_RELAXED, __HIP_MEMORY_SCOPE_AGENT)) == call ||
        __builtin_amdgcn_s_memrealtime() - t0 > DEADLINE_TICKS) {
        __hip_atomic_store(tmo, call, __ATOMIC_RELAXED, __HIP_MEMORY_SCOPE_AGENT);
        return true;
    }
    return false;
}

__device__ __forceinline__ bool gwait_all_ge(unsigned* word, bool sel, unsigned target, unsigned* tmo, unsigned call) {
    unsigned long long t0 = __builtin_amdgcn_s_memrealtime();
    for (;;) {
        const unsigned v = __hip_atomic_load(word, __ATOMIC_RELAXED, __HIP_MEMORY_SCOPE_AGENT);
        if (__all(!sel || v >= target)) return true;
        if (give_up(t0, tmo, call)) return false;
        __builtin_amdgcn_s_sleep(1);
    }
}

#define RSRC(ptr, bytes) __builtin_amdgcn_make_buffer_rsrc((void*)(ptr), 0, (bytes), 0x00020000)
#define BLOAD(rs, voff, soff, aux) __builtin_amdgcn_raw_buffer_load_b128(rs, voff, soff, aux)

__device__ __forceinline__ h8 cvt8(f4 a, f4 b) {
    h8 r = {(_Float16)a[0], (_Float16)a[1], (_Float16)a[2], (_Float16)a[3], (_Float16)b[0], (_Float16)b[1], (_Float16)b[2], (_Float16)b[3]};
    return r;
}

template <int NU, int WIN, class F>
__device__ __forceinline__ void load_w_units(__amdgpu_buffer_rsrc_t wrs, unsigned voff, unsigned soff0, unsigned char* ldsw, int lane, F after_issue) {
    f4 t[WIN][8];
#define W_ISSUE(u) _Pragma("unroll") for (int jj = 0; jj < 8; ++jj) \
        t[(u) % WIN][jj] = __builtin_bit_cast(f4, BLOAD(wrs, voff, soff0 + (unsigned)((u) * 32 + (jj & 3) + 16 * (jj >> 2)) * (NN * 4u), WPOL))
#pragma unroll
    for (int u = 0; u < WIN; ++u) { W_ISSUE(u); }
    __builtin_amdgcn_sched_barrier(0);
    after_issue();
    __builtin_amdgcn_sched_barrier(0);
#pragma unroll
    for (int u = 0; u < NU; ++u) {
        h8 f[4];
#pragma unroll
        for (int tt = 0; tt < 4; ++tt)
#pragma unroll
            for (int jj = 0; jj < 8; ++jj) f[tt][jj] = (_Float16)t[u % WIN][jj][tt];
        __builtin_amdgcn_sched_barrier(0);
        if (u + WIN < NU) { W_ISSUE(u + WIN); }
#pragma unroll
        for (int tt = 0; tt < 4; ++tt) *(h8*)(ldsw + (u * 8 + tt) * 1024 + lane * 16) = f[tt];
        __builtin_amdgcn_sched_barrier(0);
    }
#undef W_ISSUE
}

__device__ __forceinline__ int pass_m(int q) { return q < 2 ? q : (q < 14 ? 2 + ((q - 2) >> 1) : 15 - q); }
__device__ __forceinline__ int pass_h(int q) { return q < 2 ? 0 : (q < 14 ? (q & 1) : 1); }

typedef __attribute__((address_space(3))) const h8 lds_h8;
__device__ __forceinline__ h8 lds_frag(unsigned addr, int imm) { return *(lds_h8*)(size_t)(addr + imm); }

struct Fin {
    __amdgpu_buffer_rsrc_t wsrs;
    unsigned char* myws;
    unsigned* tmo;
    lds_u32* cdone;
    unsigned pvoff, dvoff0, seq0, call;
    int lane;
    bool same;
};

__device__ __forceinline__ void ensure_a(h8 (&a)[16], __amdgpu_buffer_rsrc_t wsrs, unsigned avoff, unsigned soff, unsigned* tmo, unsigned call) {
    unsigned long long t0 = 0;
    for (int tries = 0;; ++tries) {
        unsigned m = ~0u;
#pragma unroll
        for (int s = 0; s < 16; ++s) { const u4 w = __builtin_bit_cast(u4, a[s]); m = min(m, min(w[0], w[2])); }
        if (__all(m != 0u)) return;
        if (tries == 0) t0 = __builtin_amdgcn_s_memrealtime();
        if (give_up(t0, tmo, call)) return;
        asm volatile("" ::: "memory");
#pragma unroll
        for (int s = 0; s < 16; ++s) a[s] = __builtin_bit_cast(h8, BLOAD(wsrs, avoff, soff + s * 1024, 16));
    }
}

__device__ __forceinline__ void publish(const Fin& f, int ps, const f4 (&acc)[4], unsigned dv) {
    if (ps >= RING) {
        const unsigned need = f.seq0 + (ps - RING + 1);
        if (!__all(dv >= need)) gwait_all_ge((unsigned*)(f.myws + f.dvoff0 + (ps % RING) * 4u), true, need, f.tmo, f.call);
    }
    const unsigned so = (unsigned)(ps % RING) * SLOT_BYTES;
    const unsigned par = tag_of(ps, f.call);
#pragma unroll
    for (int jr = 0; jr < 4; ++jr) {
        u4 v = {(__float_as_uint(acc[0][jr]) & ~TAG_MASK) | par, __float_as_uint(acc[1][jr]), (__float_as_uint(acc[2][jr]) & ~TAG_MASK) | par, __float_as_uint(acc[3][jr])};
        if (f.same) __builtin_amdgcn_raw_buffer_store_b128(v, f.wsrs, f.pvoff, so + jr * 512, 0);
        else __builtin_amdgcn_raw_buffer_store_b128(v, f.wsrs, f.pvoff, so + jr * 512, 16);
    }
    if (f.lane == 0) __hip_atomic_fetch_add(f.cdone, 1u, __ATOMIC_RELAXED, __HIP_MEMORY_SCOPE_WORKGROUP);
}

template <bool RELOAD, bool PREV>
__device__ __forceinline__ void pass_body(int ps, h8 (&a)[16], f4 (&acc)[4], const f4 (&accp)[4], unsigned dvp, unsigned& dv, h8 (&b)[3][4],
                                          unsigned wb, unsigned wb_next, const Fin& f, unsigned avoff, unsigned nsoff, unsigned csoff) {
    unsigned amin = ~0u;
    unsigned lo = wb, hi = wb + 65536u;
    asm volatile("" : "+v"(lo), "+v"(hi));
#pragma unroll
    for (int s = 0; s < 16; ++s) {
#pragma unroll
        for (int t = 0; t < 4; ++t) {
            if (s == 0) acc[t] = __builtin_amdgcn_mfma_f32_16x16x32_f16(a[s], b[0][t], (f4){0.f, 0.f, 0.f, 0.f}, 0, 0, 0);
            else acc[t] = __builtin_amdgcn_mfma_f32_16x16x32_f16(a[s], b[s % 3][t], acc[t], 0, 0, 0);
            if (s + 2 < 8) b[(s + 2) % 3][t] = lds_frag(lo, ((s + 2) * 8 + t) * 1024);
            else if (s + 2 < 16) b[(s + 2) % 3][t] = lds_frag(hi, ((s + 2 - 8) * 8 + t) * 1024);
            __builtin_amdgcn_sched_barrier(0);
        }
        if (s == 0) {
            if (PREV) publish(f, ps - 1, accp, dvp);
            asm volatile("" ::: "memory");
            dv = __builtin_amdgcn_raw_buffer_load_b32(f.wsrs, f.dvoff0, (ps % RING) * 4u, 16);
            asm volatile("" ::: "memory");
        }
        { const u4 w = __builtin_bit_cast(u4, a[s]); amin = min(amin, min(w[0], w[2])); }
        if (RELOAD) a[s] = __builtin_bit_cast(h8, BLOAD(f.wsrs, avoff, nsoff + s * 1024, APOL));
        __builtin_amdgcn_sched_barrier(0);
    }
    if (!__all(amin != 0u)) {
#pragma unroll
        for (int t = 0; t < 4; ++t) acc[t] = (f4){0.f, 0.f, 0.f, 0.f};
        unsigned long long t0 = __builtin_amdgcn_s_memrealtime();
#pragma unroll 1
        for (int q = 0; q < 4; ++q) {
            u4 w[4];
            for (;;) {
                asm volatile("" ::: "memory");
                unsigned m = ~0u;
#pragma unroll
                for (int i = 0; i < 4; ++i) w[i] = BLOAD(f.wsrs, avoff, csoff + (q * 4 + i) * 1024, 16);
#pragma unroll
                for (int i = 0; i < 4; ++i) m = min(m, min(w[i][0], w[i][2]));
                if (__all(m != 0u)) break;
                if (give_up(t0, f.tmo, f.call)) break;
            }
#pragma unroll
            for (int i = 0; i < 4; ++i)
#pragma unroll
                for (int t = 0; t < 4; ++t)
                    acc[t] = __builtin_amdgcn_mfma_f32_16x16x32_f16(__builtin_bit_cast(h8, w[i]), lds_frag(wb + (q * 4 + i) * 8192u, t * 1024), acc[t], 0, 0, 0);
        }
    }
    unsigned nx = wb_next;
    asm volatile("" : "+v"(nx));
#pragma unroll
    for (int s = 0; s < 2; ++s)
#pragma unroll
        for (int t = 0; t < 4; ++t) b[s][t] = lds_frag(nx, (s * 8 + t) * 1024);
}

template <class F>
__device__ __forceinline__ void prepass(const Params& p, __amdgpu_buffer_rsrc_t wsrs, lds_u32* lf, int idx, int j, int q4, int lane, F after_issue) {
    const int c = lane & 15, g = lane >> 4;
    const auto xrs = RSRC(p.x, ML * KK * 4u);
    const unsigned voff = ((unsigned)(idx * 16 + c) * KK + j * 512 + q4 * 128 + g * 4) * 4u;
    f4 t[8];
#pragma unroll
    for (int q = 0; q < 4; ++q) {
        t[2 * q] = __builtin_bit_cast(f4, BLOAD(xrs, voff + q * 128, 0, XPOL));
        t[2 * q + 1] = __builtin_bit_cast(f4, BLOAD(xrs, voff + q * 128 + 64, 0, XPOL));
    }
    __builtin_amdgcn_sched_barrier(0);
    after_issue();
    __builtin_amdgcn_sched_barrier(0);
    u4 f[4];
#pragma unroll
    for (int q = 0; q < 4; ++q) {
        f[q] = __builtin_bit_cast(u4, cvt8(t[2 * q], t[2 * q + 1]));
        if (f[q][0] == 0u) f[q][0] = 0x8000u;
        if (f[q][2] == 0u) f[q][2] = 0x8000u;
    }
    lds_wait_ge(&lf[F_CALL], 1);
    const unsigned call = __builtin_amdgcn_readfirstlane(lds_load(&lf[F_CALL]));
    const unsigned xo = OFF_X16 + (call & 1u) * (4u << 20) + (unsigned)((idx * 8 + j) * 16 + q4 * 4) * 1024u + lane * 16u;
#pragma unroll
    for (int q = 0; q < 4; ++q) __builtin_amdgcn_raw_buffer_store_b128(f[q], wsrs, xo + q * 1024, 0, 16);
}

__global__ __launch_bounds__(NTH, 2) void fused_kernel(Params p) {
    const unsigned long long tstart = __builtin_amdgcn_s_memrealtime();
    __shared__ __attribute__((aligned(16))) unsigned char smem[LDS_BYTES];
    const int tid = threadIdx.x, lane = tid & 63;
    const int wave = __builtin_amdgcn_readfirstlane(tid >> 6);
    const int bid = blockIdx.x;
    const int rank = p.rank;
    const int j = bid & 7, idx = bid >> 3;
    const int dest = j;
    const int kq = idx & 7, nb = idx >> 3;
    const int grp = j * 4 + nb;
    unsigned char* myws = p.ws[rank];
    lds_u32* lf = (lds_u32*)(smem + LDS_FLAG);
    unsigned ctr = 0;
    if (wave == 0) ctr = __hip_atomic_load((unsigned*)(myws + OFF_CTR) + bid, __ATOMIC_RELAXED, __HIP_MEMORY_SCOPE_AGENT);
    if (tid < F_NWORDS) lf[tid] = 0;
    asm volatile("s_waitcnt lgkmcnt(0)" ::: "memory");
    __builtin_amdgcn_s_barrier();
    unsigned* tmo = (unsigned*)(myws + OFF_TMO);
    const auto wsrs = RSRC(myws, (unsigned)WS_USED);
    const int c = lane & 15, g = lane >> 4;

    if (wave < 4) {
        prepass(p, wsrs, lf, idx, j, wave, lane, [&]() {
            if (wave == 0) {
                const unsigned call0 = __builtin_amdgcn_readfirstlane(ctr) + 1;
                if (lane == 0) {
                    __hip_atomic_store(&lf[F_CALL], call0, __ATOMIC_RELAXED, __HIP_MEMORY_SCOPE_WORKGROUP);
                    unsigned xcc;
                    asm volatile("s_getreg_b32 %0, hwreg(HW_REG_XCC_ID, 0, 4)" : "=s"(xcc));
                    __hip_atomic_store((unsigned*)(myws + OFF_XCC + grp * 128 + kq * 4), (call0 << 4) | xcc, __ATOMIC_RELAXED, __HIP_MEMORY_SCOPE_AGENT);
                    __hip_atomic_store((unsigned*)(myws + OFF_CTR) + bid, call0, __ATOMIC_RELAXED, __HIP_MEMORY_SCOPE_AGENT);
                }
            }
        });
        const unsigned call = __builtin_amdgcn_readfirstlane(lds_load(&lf[F_CALL]));
        const unsigned avoff = OFF_X16 + (call & 1u) * (4u << 20) + (unsigned)((wave * 8 + kq) * 16) * 1024u + lane * 16u;
        h8 a[2][16];
        unsigned xcc_v;
        {
            asm volatile("s_waitcnt vmcnt(0)" ::: "memory");
            while (__builtin_amdgcn_s_memrealtime() - tstart < AT) __builtin_amdgcn_s_sleep(1);
            unsigned* tab = (unsigned*)(myws + OFF_XCC + grp * 128) + (lane & 7);
            xcc_v = __hip_atomic_load(tab, __ATOMIC_RELAXED, __HIP_MEMORY_SCOPE_AGENT);
#pragma unroll
            for (int q = 0; q < 2; ++q)
#pragma unroll
                for (int s = 0; s < 16; ++s) a[q][s] = __builtin_bit_cast(h8, BLOAD(wsrs, avoff, q * (512u << 10) + s * 1024, 16));
            if (lane == 0) __hip_atomic_fetch_add(&lf[F_AISS], 1u, __ATOMIC_RELAXED, __HIP_MEMORY_SCOPE_WORKGROUP);
            if (!__all((xcc_v >> 4) >= call)) {
                gwait_all_ge(tab, true, call << 4, tmo, call);
                xcc_v = __hip_atomic_load(tab, __ATOMIC_RELAXED, __HIP_MEMORY_SCOPE_AGENT);
            }
            ensure_a(a[0], wsrs, avoff, 0u, tmo, call);
        }
        const unsigned wb0 = (unsigned)(size_t)(lds_h8*)(smem + LDS_W) + lane * 16;
        Fin f;
        f.wsrs = wsrs; f.myws = myws; f.tmo = tmo; f.cdone = &lf[F_CDONE]; f.call = call; f.lane = lane;
        f.dvoff0 = OFF_DONE + (unsigned)(grp * 8 + 2 * wave + (lane & 1)) * 128u;
        f.seq0 = (call - 1) * NPASS;
        f.pvoff = OFF_PART + (unsigned)((grp * 8 + 2 * wave + (lane >> 5)) * 8 + kq) * 2048u + (lane & 31) * 16u;
        f.same = __all(xcc_v == (unsigned)__builtin_amdgcn_readfirstlane(xcc_v));
        if (wave == 0 && lane == 0) __hip_atomic_store(&lf[F_SAME], f.same ? 1u : 2u, __ATOMIC_RELAXED, __HIP_MEMORY_SCOPE_WORKGROUP);
        lds_wait_ge(&lf[F_WA], 4);
        f4 acc[2][4];
        unsigned dv[2] = {0u, 0u};
        h8 b[3][4];
#pragma unroll
        for (int s = 0; s < 2; ++s)
#pragma unroll
            for (int t = 0; t < 4; ++t) b[s][t] = lds_frag(wb0, (s * 8 + t) * 1024);
        auto bo = [&](int m) { return (unsigned)m * (512u << 10); };
        const unsigned wbA = wb0, wbB = wb0 + 4096;
        pass_body<false, false>(0, a[0], acc[0], acc[1], 0u, dv[0], b, wbA, wbA, f, avoff, 0u, bo(0));
        publish(f, 0, acc[0], dv[0]);
        pass_body<false, false>(1, a[1], acc[1], acc[0], 0u, dv[1], b, wbA, wbA, f, avoff, 0u, bo(1));
        publish(f, 1, acc[1], dv[1]);
#pragma unroll
        for (int q = 0; q < 2; ++q)
#pragma unroll
            for (int s = 0; s < 16; ++s) a[q][s] = __builtin_bit_cast(h8, BLOAD(wsrs, avoff, bo(2 + q) + s * 1024, APOL));
        lds_wait_ge(&lf[F_WB], NWB);
        pass_body<false, false>(2, a[0], acc[0], acc[1], 0u, dv[0], b, wbA, wbB, f, avoff, 0u, bo(2));
        pass_body<true, true>(3, a[0], acc[1], acc[0], dv[0], dv[1], b, wbB, wbA, f, avoff, bo(4), bo(2));
        pass_body<false, true>(4, a[1], acc[0], acc[1], dv[1], dv[0], b, wbA, wbB, f, avoff, 0u, bo(3));
        pass_body<true, true>(5, a[1], acc[1], acc[0], dv[0], dv[1], b, wbB, wbA, f, avoff, bo(5), bo(3));
        for (int m = 4; m < 8; m += 2) {
            const int ps = 2 * m - 2;
            pass_body<false, true>(ps, a[0], acc[0], acc[1], dv[1], dv[0], b, wbA, wbB, f, avoff, 0u, bo(m));
            pass_body<true, true>(ps + 1, a[0], acc[1], acc[0], dv[0], dv[1], b, wbB, wbA, f, avoff, bo(m == 4 ? 6 : 1), bo(m));
            pass_body<false, true>(ps + 2, a[1], acc[0], acc[1], dv[1], dv[0], b, wbA, wbB, f, avoff, 0u, bo(m + 1));
            pass_body<true, true>(ps + 3, a[1], acc[1], acc[0], dv[0], dv[1], b, wbB, m == 4 ? wbA : wbB, f, avoff, bo(m == 4 ? 7 : 0), bo(m + 1));
        }
        pass_body<false, true>(14, a[0], acc[0], acc[1], dv[1], dv[0], b, wbB, wbB, f, avoff, 0u, bo(1));
        pass_body<false, true>(15, a[1], acc[1], acc[0], dv[0], dv[1], b, wbB, wbB, f, avoff, 0u, bo(0));
        publish(f, NPASS - 1, acc[1], dv[1]);
        {
            const unsigned xz = OFF_X16 + ((call + 1) & 1u) * (4u << 20) + (unsigned)((idx * 8 + j) * 16 + wave * 4) * 1024u + lane * 16u;
            const u4 z = {0u, 0u, 0u, 0u};
#pragma unroll
            for (int q = 0; q < 4; ++q) __builtin_amdgcn_raw_buffer_store_b128(z, wsrs, xz + q * 1024, 0, 16);
        }
    } else {
        const auto wrs = RSRC(p.w, (unsigned)KK * NN * 4u);
        const unsigned wvoff = ((unsigned)(kq * 512 + g * 4) * NN + dest * NB + nb * 128 + c * 4) * 4u;
        if (WDEL) __builtin_amdgcn_s_sleep(WDEL);
        {
            const int u = wave - 4;
            load_w_units<4, 4>(wrs, wvoff, (unsigned)(u * 128) * (NN * 4u), smem + LDS_W + u * 32768, lane, []() {});
            lds_add1(&lf[F_WA], lane);
        }
        if (wave != 6) {
            lds_wait_ge(&lf[F_AISS], 4);
            if (wave == 5) load_w_units<6, 6>(wrs, wvoff + 256, 0u, smem + LDS_W + 4096, lane, []() {});
            else {
                const int s0 = wave == 4 ? 6 : 11;
                load_w_units<5, 5>(wrs, wvoff + 256, (unsigned)(s0 * 32) * (NN * 4u), smem + LDS_W + 4096 + s0 * 8192, lane, []() {});
            }
            lds_add1(&lf[F_WB], lane);
        }
        if (wave < 7) {
            lds_wait_ge(&lf[F_CALL], 1);
            const unsigned call = __builtin_amdgcn_readfirstlane(lds_load(&lf[F_CALL]));
            lds_wait_ge(&lf[F_SAME], 1);
            const bool same = __builtin_amdgcn_readfirstlane(lds_load(&lf[F_SAME])) == 1;
            const unsigned dnoff = OFF_DONE + (unsigned)(grp * 8 + kq) * 128u;
            const unsigned seq0 = (call - 1) * NPASS;
            const unsigned pb0 = OFF_PART + (unsigned)((grp * 8 + kq) * 8) * 2048u + lane * 16u;
            for (;;) {
                const int ps = (int)lds_ticket(&lf[F_RTICKET]);
                if (ps >= NPASS) break;
                lds_wait_ge(&lf[F_CDONE], 4 * (ps + 1));
                const unsigned so = (unsigned)(ps % RING) * SLOT_BYTES;
                const unsigned par = tag_of(ps, call);
                u4 v[8][2];
                bool bad = false;
                unsigned long long t0 = 0;
                unsigned pend = 0xffu;
                unsigned tv = 0;
                for (int tries = 0;; ++tries) {
#pragma unroll
                    for (int pr = 0; pr < 8; ++pr)
                        if (pend & (1u << pr)) {
                            asm volatile("" ::: "memory");
                            v[pr][0] = BLOAD(wsrs, pb0, so + pr * 2048, RPOL);
                            v[pr][1] = BLOAD(wsrs, pb0 + 1024, so + pr * 2048, RPOL);
                        }
                    tv = __hip_atomic_load(tmo, __ATOMIC_RELAXED, __HIP_MEMORY_SCOPE_AGENT);
                    unsigned np = 0;
#pragma unroll
                    for (int pr = 0; pr < 8; ++pr) {
                        const unsigned m = ((v[pr][0][0] & v[pr][0][2] & v[pr][1][0] & v[pr][1][2]) & TAG_MASK) == par &&
                                           ((v[pr][0][0] | v[pr][0][2] | v[pr][1][0] | v[pr][1][2]) & TAG_MASK) == par;
                        if (!__all(m)) np |= 1u << pr;
                    }
                    pend = np;
                    if (!pend) break;
                    if (tries == 0) t0 = __builtin_amdgcn_s_memrealtime();
                    if ((unsigned)__builtin_amdgcn_readfirstlane(tv) == call || __builtin_amdgcn_s_memrealtime() - t0 > DEADLINE_TICKS) {
                        __hip_atomic_store(tmo, call, __ATOMIC_RELAXED, __HIP_MEMORY_SCOPE_AGENT);
                        bad = true;
                        break;
                    }
                    __builtin_amdgcn_s_sleep(2);
                }
                if (lane == 0) {
                    if (same) __builtin_amdgcn_raw_buffer_store_b32(seq0 + ps + 1, wsrs, dnoff + (ps % RING) * 4u, 0, 0);
                    else __builtin_amdgcn_raw_buffer_store_b32(seq0 + ps + 1, wsrs, dnoff + (ps % RING) * 4u, 0, 16);
                }
                f4 o[2];
#pragma unroll
                for (int q = 0; q < 2; ++q) {
                    o[q] = __builtin_bit_cast(f4, v[0][q]);
#pragma unroll
                    for (int pr = 1; pr < 8; ++pr) o[q] += __builtin_bit_cast(f4, v[pr][q]);
                }
                if (ps >= ORING) lds_wait_ge(&lf[F_OFREE], ps - ORING + 1);
                f4 r[2];
#pragma unroll
                for (int q = 0; q < 2; ++q) r[q] = (f4){silu(o[q][0]), silu(o[q][1]), silu(o[q][2]), silu(o[q][3])};
                bad = bad || ((unsigned)__builtin_amdgcn_readfirstlane(tv) == call);
#pragma unroll
                for (int q = 0; q < 2; ++q) {
                    if (bad) r[q] = (f4){__builtin_nanf(""), __builtin_nanf(""), __builtin_nanf(""), __builtin_nanf("")};
                    *(f4*)(smem + LDS_OUT + (ps % ORING) * 2048 + q * 1024 + lane * 16) = r[q];
                }
                asm volatile("s_waitcnt lgkmcnt(0)" ::: "memory");
                if (lane == 0) __hip_atomic_store(&lf[F_ORDY + ps], 1u, __ATOMIC_RELAXED, __HIP_MEMORY_SCOPE_WORKGROUP);
            }
        } else {
            lds_wait_ge(&lf[F_CALL], 1);
            const unsigned call = __builtin_amdgcn_readfirstlane(lds_load(&lf[F_CALL]));
            float* obase = p.out[dest] + (size_t)(rank * ML + kq * 8 + ((lane >> 4) & 1) * 4 + (lane >> 5)) * NB + nb * 128 + c * 4;
            for (int ps = 0; ps < NPASS; ++ps) {
                lds_wait_ge(&lf[F_ORDY + ps], 1);
                f4 r[2];
#pragma unroll
                for (int q = 0; q < 2; ++q) r[q] = *(const f4*)(smem + LDS_OUT + (ps % ORING) * 2048 + q * 1024 + lane * 16);
                asm volatile("s_waitcnt lgkmcnt(0)" ::: "memory");
                if (lane == 0) __hip_atomic_store(&lf[F_OFREE], (unsigned)(ps + 1), __ATOMIC_RELAXED, __HIP_MEMORY_SCOPE_WORKGROUP);
                float* o = obase + (size_t)(pass_m(ps) * 64) * NB + pass_h(ps) * 64;
#pragma unroll
                for (int q = 0; q < 2; ++q)
                    asm volatile("global_store_dwordx4 %0, %1, off sc0 sc1\n\ts_nop 1" :: "v"(o + q * 2 * NB), "v"(r[q]) : "memory");
            }
            asm volatile("s_waitcnt vmcnt(0)" ::: "memory");
            if (lane == 0) {
                if (dest != rank) {
                    __hip_atomic_fetch_add((unsigned*)(p.ws[dest] + OFF_RFLAG + rank * 128), 1u, __ATOMIC_RELAXED, __HIP_MEMORY_SCOPE_SYSTEM);
                } else if (idx < NR && idx != rank) {
                    unsigned* flag = (unsigned*)(myws + OFF_RFLAG + idx * 128);
                    const unsigned target = call * 32u;
                    unsigned long long t0 = __builtin_amdgcn_s_memrealtime();
                    while (__hip_atomic_load(flag, __ATOMIC_RELAXED, __HIP_MEMORY_SCOPE_SYSTEM) < target) {
                        if (__hip_atomic_load(tmo, __ATOMIC_RELAXED, __HIP_MEMORY_SCOPE_AGENT) == call ||
                            __builtin_amdgcn_s_memrealtime() - t0 > DEADLINE_TICKS) {
                            __hip_atomic_store(tmo, call, __ATOMIC_RELAXED, __HIP_MEMORY_SCOPE_AGENT);
                            obase[0] = __builtin_nanf("");
                            break;
                        }
                        __builtin_amdgcn_s_sleep(2);
                    }
                    __builtin_amdgcn_fence(__ATOMIC_ACQUIRE, "");
                }
            }
        }
    }
}

extern "C" void kernel_launch(int rank, int nranks, ncclComm_t comm,
                              hipStream_t stream,
                              void* const* d_in, const int* in_sizes, int n_in,
                              void* d_out, int out_size,
                              const int* in_dtypes, int out_dtype,
                              void* const* d_out_all,
                              void* const* d_ws_all, size_t ws_bytes) {
    if (nranks != NR || ws_bytes < WS_USED || n_in < 2 || rank < 0 || rank >= NR) { fprintf(stderr, "bad config\n"); return; }
    Params p;
    p.x = (const float*)d_in[0];
    p.w = (const float*)d_in[1];
    for (int i = 0; i < NR; ++i) { p.out[i] = (float*)d_out_all[i]; p.ws[i] = (unsigned char*)d_ws_all[i]; }
    p.rank = rank;
    fused_kernel<<<NWG, NTH, 0, stream>>>(p);
    HIP_CHECK(hipGetLastError());
}
```

```cpp
#include <hip/hip_runtime.h>
#include <rccl/rccl.h>
#include <cstdio>
#include <cstdint>

#define HIP_CHECK(expr) do { hipError_t _e = (expr); if (_e != hipSuccess) { \
    fprintf(stderr, "HIP error %s at %s:%d\n", hipGetErrorString(_e), __FILE__, __LINE__); \
    return; } } while (0)

typedef _Float16 h8 __attribute__((ext_vector_type(8)));
typedef float f4 __attribute__((ext_vector_type(4)));
typedef unsigned u4 __attribute__((ext_vector_type(4)));

constexpr int APOL = 16;
constexpr int RPOL = 16;
constexpr int WPOL = 2;
constexpr int XPOL = 2;
constexpr int NWB = 3;
constexpr unsigned long long AT = 600;
constexpr int DVS = 8;

constexpr int NR = 8;
constexpr int ML = 512;
constexpr int KK = 4096;
constexpr int NN = 4096;
constexpr int NB = 512;
constexpr int NWG = 256;
constexpr int NTH = 512;
constexpr int NPASS = 16;
constexpr int RING = 2;
constexpr int ORING = 8;
constexpr unsigned SLOT_BYTES = 4u << 20;
constexpr unsigned long long DEADLINE_TICKS = 50000000ull;
constexpr unsigned TAG_MASK = 7u;
__device__ __forceinline__ unsigned tag_of(int ps, unsigned call) {
    const int r = ps % RING;
    const unsigned uses = (unsigned)((NPASS - 1 - r) / RING + 1);
    const unsigned n = (call - 1) * uses + (unsigned)(ps / RING) + 1u;
    return (n % 7u) + 1u;
}

constexpr unsigned OFF_TMO = 128;
constexpr unsigned OFF_RFLAG = 256;
constexpr unsigned OFF_CTR = 3072;
constexpr unsigned OFF_XCC = 4096;
constexpr unsigned OFF_DONE = 8192;
constexpr unsigned OFF_X16 = 1u << 20;
constexpr unsigned OFF_PART = 9u << 20;
constexpr size_t WS_USED = (size_t)OFF_PART + (size_t)RING * SLOT_BYTES;

constexpr int LDS_W = 0;
constexpr int LDS_OUT = 131072;
constexpr int LDS_FLAG = LDS_OUT + ORING * 2048;
constexpr int LDS_BYTES = LDS_FLAG + 256;
enum { F_CALL = 0, F_WA, F_WB, F_AISS, F_CDONE, F_RTICKET, F_OFREE, F_SAME, F_ORDY  , F_NWORDS = F_ORDY + 16 };

struct Params {
    const float* x;
    const float* w;
    float* out[NR];
    unsigned char* ws[NR];
    int rank;
};

__device__ __forceinline__ float silu(float y) {
    float c = fminf(fmaxf(y, -60.0f), 60.0f);
    return y * __builtin_amdgcn_rcpf(1.0f + __expf(-c));
}

typedef __attribute__((address_space(3))) unsigned lds_u32;
__device__ __forceinline__ unsigned lds_load(lds_u32* p) { return __hip_atomic_load(p, __ATOMIC_RELAXED, __HIP_MEMORY_SCOPE_WORKGROUP); }
__device__ __forceinline__ void lds_add1(lds_u32* p, int lane) {
    asm volatile("s_waitcnt lgkmcnt(0)" ::: "memory");
    if (lane == 0) __hip_atomic_fetch_add(p, 1u, __ATOMIC_RELAXED, __HIP_MEMORY_SCOPE_WORKGROUP);
}

__device__ __forceinline__ unsigned lds_ticket(lds_u32* p) {
    unsigned old;
    unsigned long long save;
    const unsigned addr = (unsigned)(size_t)p, one = 1u;
    asm volatile("s_mov_b64 %1, exec\n\ts_mov_b64 exec, 1\n\tds_add_rtn_u32 %0, %2, %3\n\ts_waitcnt lgkmcnt(0)\n\ts_mov_b64 exec, %1"
                 : "=&v"(old), "=&s"(save) : "v"(addr), "v"(one) : "memory");
    return __builtin_amdgcn_readfirstlane(old);
}

__device__ __forceinline__ bool lds_wait_ge(lds_u32* p, unsigned target) {
    if ((unsigned)__builtin_amdgcn_readfirstlane(lds_load(p)) >= target) return true;
    unsigned long long t0 = __builtin_amdgcn_s_memrealtime();
    while ((unsigned)__builtin_amdgcn_readfirstlane(lds_load(p)) < target) {
        if (__builtin_amdgcn_s_memrealtime() - t0 > 2 * DEADLINE_TICKS) return false;
        __builtin_amdgcn_s_sleep(1);
    }
    return true;
}

__device__ __forceinline__ bool give_up(unsigned long long t0, unsigned* tmo, unsigned call) {
    if ((unsigned)__builtin_amdgcn_readfirstlane(__hip_atomic_load(tmo, __ATOMIC_RELAXED, __HIP_MEMORY_SCOPE_AGENT)) == call ||
        __builtin_amdgcn_s_memrealtime() - t0 > DEADLINE_TICKS) {
        __hip_atomic_store(tmo, call, __ATOMIC_RELAXED, __HIP_MEMORY_SCOPE_AGENT);
        return true;
    }
    return false;
}

__device__ __forceinline__ bool gwait_all_ge(unsigned* word, bool sel, unsigned target, unsigned* tmo, unsigned call) {
    unsigned long long t0 = __builtin_amdgcn_s_memrealtime();
    for (;;) {
        const unsigned v = __hip_atomic_load(word, __ATOMIC_RELAXED, __HIP_MEMORY_SCOPE_AGENT);
        if (__all(!sel || v >= target)) return true;
        if (give_up(t0, tmo, call)) return false;
        __builtin_amdgcn_s_sleep(1);
    }
}

#define RSRC(ptr, bytes) __builtin_amdgcn_make_buffer_rsrc((void*)(ptr), 0, (bytes), 0x00020000)
#define BLOAD(rs, voff, soff, aux) __builtin_amdgcn_raw_buffer_load_b128(rs, voff, soff, aux)

__device__ __forceinline__ h8 cvt8(f4 a, f4 b) {
    h8 r = {(_Float16)a[0], (_Float16)a[1], (_Float16)a[2], (_Float16)a[3], (_Float16)b[0], (_Float16)b[1], (_Float16)b[2], (_Float16)b[3]};
    return r;
}

template <int NU, int WIN, class F>
__device__ __forceinline__ void load_w_units(__amdgpu_buffer_rsrc_t wrs, unsigned voff, unsigned soff0, unsigned char* ldsw, int lane, F after_issue) {
    f4 t[WIN][8];
#define W_ISSUE(u) _Pragma("unroll") for (int jj = 0; jj < 8; ++jj) \
        t[(u) % WIN][jj] = __builtin_bit_cast(f4, BLOAD(wrs, voff, soff0 + (unsigned)((u) * 32 + (jj & 3) + 16 * (jj >> 2)) * (NN * 4u), WPOL))
#pragma unroll
    for (int u = 0; u < WIN; ++u) { W_ISSUE(u); }
    __builtin_amdgcn_sched_barrier(0);
    after_issue();
    __builtin_amdgcn_sched_barrier(0);
#pragma unroll
    for (int u = 0; u < NU; ++u) {
        h8 f[4];
#pragma unroll
        for (int tt = 0; tt < 4; ++tt)
#pragma unroll
            for (int jj = 0; jj < 8; ++jj) f[tt][jj] = (_Float16)t[u % WIN][jj][tt];
        __builtin_amdgcn_sched_barrier(0);
        if (u + WIN < NU) { W_ISSUE(u + WIN); }
#pragma unroll
        for (int tt = 0; tt < 4; ++tt) *(h8*)(ldsw + (u * 8 + tt) * 1024 + lane * 16) = f[tt];
        __builtin_amdgcn_sched_barrier(0);
    }
#undef W_ISSUE
}

__device__ __forceinline__ int pass_m(int q) { return q < 2 ? q : (q < 14 ? 2 + ((q - 2) >> 1) : 15 - q); }
__device__ __forceinline__ int pass_h(int q) { return q < 2 ? 0 : (q < 14 ? (q & 1) : 1); }

typedef __attribute__((address_space(3))) const h8 lds_h8;
__device__ __forceinline__ h8 lds_frag(unsigned addr, int imm) { return *(lds_h8*)(size_t)(addr + imm); }

struct Fin {
    __amdgpu_buffer_rsrc_t wsrs;
    unsigned char* myws;
    unsigned* tmo;
    lds_u32* cdone;
    unsigned pvoff, dvoff0, seq0, call;
    int lane;
    bool same;
};

__device__ __forceinline__ void ensure_a(h8 (&a)[16], __amdgpu_buffer_rsrc_t wsrs, unsigned avoff, unsigned soff, unsigned* tmo, unsigned call) {
    unsigned long long t0 = 0;
    for (int tries = 0;; ++tries) {
        unsigned m = ~0u;
#pragma unroll
        for (int s = 0; s < 16; ++s) { const u4 w = __builtin_bit_cast(u4, a[s]); m = min(m, min(w[0], w[2])); }
        if (__all(m != 0u)) return;
        if (tries == 0) t0 = __builtin_amdgcn_s_memrealtime();
        if (give_up(t0, tmo, call)) return;
        asm volatile("" ::: "memory");
#pragma unroll
        for (int s = 0; s < 16; ++s) a[s] = __builtin_bit_cast(h8, BLOAD(wsrs, avoff, soff + s * 1024, 16));
    }
}

__device__ __forceinline__ void publish(const Fin& f, int ps, const f4 (&acc)[4], unsigned dv) {
    if (ps >= RING) {
        const unsigned need = f.seq0 + (ps - RING + 1);
        if (!__all(dv >= need)) gwait_all_ge((unsigned*)(f.myws + f.dvoff0 + (ps % RING) * 4u), true, need, f.tmo, f.call);
    }
    const unsigned so = (unsigned)(ps % RING) * SLOT_BYTES;
    const unsigned par = tag_of(ps, f.call);
#pragma unroll
    for (int jr = 0; jr < 4; ++jr) {
        u4 v = {(__float_as_uint(acc[0][jr]) & ~TAG_MASK) | par, __float_as_uint(acc[1][jr]), (__float_as_uint(acc[2][jr]) & ~TAG_MASK) | par, __float_as_uint(acc[3][jr])};
        if (f.same) __builtin_amdgcn_raw_buffer_store_b128(v, f.wsrs, f.pvoff, so + jr * 512, 0);
        else __builtin_amdgcn_raw_buffer_store_b128(v, f.wsrs, f.pvoff, so + jr * 512, 16);
    }
    if (f.lane == 0) __hip_atomic_fetch_add(f.cdone, 1u, __ATOMIC_RELAXED, __HIP_MEMORY_SCOPE_WORKGROUP);
}

template <bool RELOAD, bool PREV>
__device__ __forceinline__ void pass_body(int ps, h8 (&a)[16], f4 (&acc)[4], const f4 (&accp)[4], unsigned dvp, unsigned& dv, h8 (&b)[3][4],
                                          unsigned wb, unsigned wb_next, const Fin& f, unsigned avoff, unsigned nsoff, unsigned csoff) {
    unsigned amin = ~0u;
    unsigned lo = wb, hi = wb + 65536u;
    asm volatile("" : "+v"(lo), "+v"(hi));
#pragma unroll
    for (int s = 0; s < 16; ++s) {
#pragma unroll
        for (int t = 0; t < 4; ++t) {
            if (s == 0) acc[t] = __builtin_amdgcn_mfma_f32_16x16x32_f16(a[s], b[0][t], (f4){0.f, 0.f, 0.f, 0.f}, 0, 0, 0);
            else acc[t] = __builtin_amdgcn_mfma_f32_16x16x32_f16(a[s], b[s % 3][t], acc[t], 0, 0, 0);
            if (s + 2 < 8) b[(s + 2) % 3][t] = lds_frag(lo, ((s + 2) * 8 + t) * 1024);
            else if (s + 2 < 16) b[(s + 2) % 3][t] = lds_frag(hi, ((s + 2 - 8) * 8 + t) * 1024);
            __builtin_amdgcn_sched_barrier(0);
        }
        if (s == 0 && PREV) publish(f, ps - 1, accp, dvp);
        if (s == DVS) {
            asm volatile("" ::: "memory");
            dv = __builtin_amdgcn_raw_buffer_load_b32(f.wsrs, f.dvoff0, (ps % RING) * 4u, 16);
            asm volatile("" ::: "memory");
        }
        { const u4 w = __builtin_bit_cast(u4, a[s]); amin = min(amin, min(w[0], w[2])); }
        if (RELOAD) a[s] = __builtin_bit_cast(h8, BLOAD(f.wsrs, avoff, nsoff + s * 1024, APOL));
        __builtin_amdgcn_sched_barrier(0);
    }
    if (!__all(amin != 0u)) {
#pragma unroll
        for (int t = 0; t < 4; ++t) acc[t] = (f4){0.f, 0.f, 0.f, 0.f};
        unsigned long long t0 = __builtin_amdgcn_s_memrealtime();
#pragma unroll 1
        for (int q = 0; q < 4; ++q) {
            u4 w[4];
            for (;;) {
                asm volatile("" ::: "memory");
                unsigned m = ~0u;
#pragma unroll
                for (int i = 0; i < 4; ++i) w[i] = BLOAD(f.wsrs, avoff, csoff + (q * 4 + i) * 1024, 16);
#pragma unroll
                for (int i = 0; i < 4; ++i) m = min(m, min(w[i][0], w[i][2]));
                if (__all(m != 0u)) break;
                if (give_up(t0, f.tmo, f.call)) break;
            }
#pragma unroll
            for (int i = 0; i < 4; ++i)
#pragma unroll
                for (int t = 0; t < 4; ++t)
                    acc[t] = __builtin_amdgcn_mfma_f32_16x16x32_f16(__builtin_bit_cast(h8, w[i]), lds_frag(wb + (q * 4 + i) * 8192u, t * 1024), acc[t], 0, 0, 0);
        }
    }
    unsigned nx = wb_next;
    asm volatile("" : "+v"(nx));
#pragma unroll
    for (int s = 0; s < 2; ++s)
#pragma unroll
        for (int t = 0; t < 4; ++t) b[s][t] = lds_frag(nx, (s * 8 + t) * 1024);
}

template <class F>
__device__ __forceinline__ void prepass(const Params& p, __amdgpu_buffer_rsrc_t wsrs, lds_u32* lf, int idx, int j, int q4, int lane, F after_issue) {
    const int c = lane & 15, g = lane >> 4;
    const auto xrs = RSRC(p.x, ML * KK * 4u);
    const unsigned voff = ((unsigned)(idx * 16 + c) * KK + j * 512 + q4 * 128 + g * 4) * 4u;
    f4 t[8];
#pragma unroll
    for (int q = 0; q < 4; ++q) {
        t[2 * q] = __builtin_bit_cast(f4, BLOAD(xrs, voff + q * 128, 0, XPOL));
        t[2 * q + 1] = __builtin_bit_cast(f4, BLOAD(xrs, voff + q * 128 + 64, 0, XPOL));
    }
    __builtin_amdgcn_sched_barrier(0);
    after_issue();
    __builtin_amdgcn_sched_barrier(0);
    u4 f[4];
#pragma unroll
    for (int q = 0; q < 4; ++q) {
        f[q] = __builtin_bit_cast(u4, cvt8(t[2 * q], t[2 * q + 1]));
        if (f[q][0] == 0u) f[q][0] = 0x8000u;
        if (f[q][2] == 0u) f[q][2] = 0x8000u;
    }
    lds_wait_ge(&lf[F_CALL], 1);
    const unsigned call = __builtin_amdgcn_readfirstlane(lds_load(&lf[F_CALL]));
    const unsigned xo = OFF_X16 + (call & 1u) * (4u << 20) + (unsigned)((idx * 8 + j) * 16 + q4 * 4) * 1024u + lane * 16u;
#pragma unroll
    for (int q = 0; q < 4; ++q) __builtin_amdgcn_raw_buffer_store_b128(f[q], wsrs, xo + q * 1024, 0, 16);
}

__global__ __launch_bounds__(NTH, 2) void fused_kernel(Params p) {
    const unsigned long long tstart = __builtin_amdgcn_s_memrealtime();
    __shared__ __attribute__((aligned(16))) unsigned char smem[LDS_BYTES];
    const int tid = threadIdx.x, lane = tid & 63;
    const int wave = __builtin_amdgcn_readfirstlane(tid >> 6);
    const int bid = blockIdx.x;
    const int rank = p.rank;
    const int j = bid & 7, idx = bid >> 3;
    const int dest = j;
    const int kq = idx & 7, nb = idx >> 3;
    const int grp = j * 4 + nb;
    unsigned char* myws = p.ws[rank];
    lds_u32* lf = (lds_u32*)(smem + LDS_FLAG);
    unsigned ctr = 0;
    if (wave == 0) ctr = __hip_atomic_load((unsigned*)(myws + OFF_CTR) + bid, __ATOMIC_RELAXED, __HIP_MEMORY_SCOPE_AGENT);
    if (tid < F_NWORDS) lf[tid] = 0;
    asm volatile("s_waitcnt lgkmcnt(0)" ::: "memory");
    __builtin_amdgcn_s_barrier();
    unsigned* tmo = (unsigned*)(myws + OFF_TMO);
    const auto wsrs = RSRC(myws, (unsigned)WS_USED);
    const int c = lane & 15, g = lane >> 4;

    if (wave < 4) {
        prepass(p, wsrs, lf, idx, j, wave, lane, [&]() {
            if (wave == 0) {
                const unsigned call0 = __builtin_amdgcn_readfirstlane(ctr) + 1;
                if (lane == 0) {
                    __hip_atomic_store(&lf[F_CALL], call0, __ATOMIC_RELAXED, __HIP_MEMORY_SCOPE_WORKGROUP);
                    unsigned xcc;
                    asm volatile("s_getreg_b32 %0, hwreg(HW_REG_XCC_ID, 0, 4)" : "=s"(xcc));
                    __hip_atomic_store((unsigned*)(myws + OFF_XCC + grp * 128 + kq * 4), (call0 << 4) | xcc, __ATOMIC_RELAXED, __HIP_MEMORY_SCOPE_AGENT);
                    __hip_atomic_store((unsigned*)(myws + OFF_CTR) + bid, call0, __ATOMIC_RELAXED, __HIP_MEMORY_SCOPE_AGENT);
                }
            }
        });
        const unsigned call = __builtin_amdgcn_readfirstlane(lds_load(&lf[F_CALL]));
        const unsigned avoff = OFF_X16 + (call & 1u) * (4u << 20) + (unsigned)((wave * 8 + kq) * 16) * 1024u + lane * 16u;
        h8 a[2][16];
        unsigned xcc_v;
        {
            asm volatile("s_waitcnt vmcnt(0)" ::: "memory");
            while (__builtin_amdgcn_s_memrealtime() - tstart < AT) __builtin_amdgcn_s_sleep(1);
            unsigned* tab = (unsigned*)(myws + OFF_XCC + grp * 128) + (lane & 7);
            xcc_v = __hip_atomic_load(tab, __ATOMIC_RELAXED, __HIP_MEMORY_SCOPE_AGENT);
#pragma unroll
            for (int q = 0; q < 2; ++q)
#pragma unroll
                for (int s = 0; s < 16; ++s) a[q][s] = __builtin_bit_cast(h8, BLOAD(wsrs, avoff, q * (512u << 10) + s * 1024, 16));
            if (lane == 0) __hip_atomic_fetch_add(&lf[F_AISS], 1u, __ATOMIC_RELAXED, __HIP_MEMORY_SCOPE_WORKGROUP);
            if (!__all((xcc_v >> 4) >= call)) {
                gwait_all_ge(tab, true, call << 4, tmo, call);
                xcc_v = __hip_atomic_load(tab, __ATOMIC_RELAXED, __HIP_MEMORY_SCOPE_AGENT);
            }
            ensure_a(a[0], wsrs, avoff, 0u, tmo, call);
        }
        const unsigned wb0 = (unsigned)(size_t)(lds_h8*)(smem + LDS_W) + lane * 16;
        Fin f;
        f.wsrs = wsrs; f.myws = myws; f.tmo = tmo; f.cdone = &lf[F_CDONE]; f.call = call; f.lane = lane;
        f.dvoff0 = OFF_DONE + (unsigned)(grp * 8 + 2 * wave + (lane & 1)) * 128u;
        f.seq0 = (call - 1) * NPASS;
        f.pvoff = OFF_PART + (unsigned)((grp * 8 + 2 * wave + (lane >> 5)) * 8 + kq) * 2048u + (lane & 31) * 16u;
        f.same = __all(xcc_v == (unsigned)__builtin_amdgcn_readfirstlane(xcc_v));
        if (wave == 0 && lane == 0) __hip_atomic_store(&lf[F_SAME], f.same ? 1u : 2u, __ATOMIC_RELAXED, __HIP_MEMORY_SCOPE_WORKGROUP);
        lds_wait_ge(&lf[F_WA], 4);
        f4 acc[2][4];
        unsigned dv[2] = {0u, 0u};
        h8 b[3][4];
#pragma unroll
        for (int s = 0; s < 2; ++s)
#pragma unroll
            for (int t = 0; t < 4; ++t) b[s][t] = lds_frag(wb0, (s * 8 + t) * 1024);
        auto bo = [&](int m) { return (unsigned)m * (512u << 10); };
        const unsigned wbA = wb0, wbB = wb0 + 4096;
        pass_body<false, false>(0, a[0], acc[0], acc[1], 0u, dv[0], b, wbA, wbA, f, avoff, 0u, bo(0));
        publish(f, 0, acc[0], dv[0]);
        pass_body<false, false>(1, a[1], acc[1], acc[0], 0u, dv[1], b, wbA, wbA, f, avoff, 0u, bo(1));
        publish(f, 1, acc[1], dv[1]);
#pragma unroll
        for (int q = 0; q < 2; ++q)
#pragma unroll
            for (int s = 0; s < 16; ++s) a[q][s] = __builtin_bit_cast(h8, BLOAD(wsrs, avoff, bo(2 + q) + s * 1024, APOL));
        lds_wait_ge(&lf[F_WB], NWB);
        pass_body<false, false>(2, a[0], acc[0], acc[1], 0u, dv[0], b, wbA, wbB, f, avoff, 0u, bo(2));
        pass_body<true, true>(3, a[0], acc[1], acc[0], dv[0], dv[1], b, wbB, wbA, f, avoff, bo(4), bo(2));
        pass_body<false, true>(4, a[1], acc[0], acc[1], dv[1], dv[0], b, wbA, wbB, f, avoff, 0u, bo(3));
        pass_body<true, true>(5, a[1], acc[1], acc[0], dv[0], dv[1], b, wbB, wbA, f, avoff, bo(5), bo(3));
        for (int m = 4; m < 8; m += 2) {
            const int ps = 2 * m - 2;
            pass_body<false, true>(ps, a[0], acc[0], acc[1], dv[1], dv[0], b, wbA, wbB, f, avoff, 0u, bo(m));
            pass_body<true, true>(ps + 1, a[0], acc[1], acc[0], dv[0], dv[1], b, wbB, wbA, f, avoff, bo(m == 4 ? 6 : 1), bo(m));
            pass_body<false, true>(ps + 2, a[1], acc[0], acc[1], dv[1], dv[0], b, wbA, wbB, f, avoff, 0u, bo(m + 1));
            pass_body<true, true>(ps + 3, a[1], acc[1], acc[0], dv[0], dv[1], b, wbB, m == 4 ? wbA : wbB, f, avoff, bo(m == 4 ? 7 : 0), bo(m + 1));
        }
        pass_body<false, true>(14, a[0], acc[0], acc[1], dv[1], dv[0], b, wbB, wbB, f, avoff, 0u, bo(1));
        pass_body<false, true>(15, a[1], acc[1], acc[0], dv[0], dv[1], b, wbB, wbB, f, avoff, 0u, bo(0));
        publish(f, NPASS - 1, acc[1], dv[1]);
        {
            const unsigned xz = OFF_X16 + ((call + 1) & 1u) * (4u << 20) + (unsigned)((idx * 8 + j) * 16 + wave * 4) * 1024u + lane * 16u;
            const u4 z = {0u, 0u, 0u, 0u};
#pragma unroll
            for (int q = 0; q < 4; ++q) __builtin_amdgcn_raw_buffer_store_b128(z, wsrs, xz + q * 1024, 0, 16);
        }
    } else {
        const auto wrs = RSRC(p.w, (unsigned)KK * NN * 4u);
        const unsigned wvoff = ((unsigned)(kq * 512 + g * 4) * NN + dest * NB + nb * 128 + c * 4) * 4u;
        {
            const int u = wave - 4;
            load_w_units<4, 4>(wrs, wvoff, (unsigned)(u * 128) * (NN * 4u), smem + LDS_W + u * 32768, lane, []() {});
            lds_add1(&lf[F_WA], lane);
        }
        if (wave != 6) {
            lds_wait_ge(&lf[F_AISS], 4);
            if (wave == 5) load_w_units<6, 6>(wrs, wvoff + 256, 0u, smem + LDS_W + 4096, lane, []() {});
            else {
                const int s0 = wave == 4 ? 6 : 11;
                load_w_units<5, 5>(wrs, wvoff + 256, (unsigned)(s0 * 32) * (NN * 4u), smem + LDS_W + 4096 + s0 * 8192, lane, []() {});
            }
            lds_add1(&lf[F_WB], lane);
        }
        if (wave < 7) {
            lds_wait_ge(&lf[F_CALL], 1);
            const unsigned call = __builtin_amdgcn_readfirstlane(lds_load(&lf[F_CALL]));
            lds_wait_ge(&lf[F_SAME], 1);
            const bool same = __builtin_amdgcn_readfirstlane(lds_load(&lf[F_SAME])) == 1;
            const unsigned dnoff = OFF_DONE + (unsigned)(grp * 8 + kq) * 128u;
            const unsigned seq0 = (call - 1) * NPASS;
            const unsigned pb0 = OFF_PART + (unsigned)((grp * 8 + kq) * 8) * 2048u + lane * 16u;
            for (;;) {
                const int ps = (int)lds_ticket(&lf[F_RTICKET]);
                if (ps >= NPASS) break;
                lds_wait_ge(&lf[F_CDONE], 4 * (ps + 1));
                const unsigned so = (unsigned)(ps % RING) * SLOT_BYTES;
                const unsigned par = tag_of(ps, call);
                u4 v[8][2];
                bool bad = false;
                unsigned long long t0 = 0;
                unsigned pend = 0xffu;
                unsigned tv = 0;
                for (int tries = 0;; ++tries) {
#pragma unroll
                    for (int pr = 0; pr < 8; ++pr)
                        if (pend & (1u << pr)) {
                            asm volatile("" ::: "memory");
                            v[pr][0] = BLOAD(wsrs, pb0, so + pr * 2048, RPOL);
                            v[pr][1] = BLOAD(wsrs, pb0 + 1024, so + pr * 2048, RPOL);
                        }
                    tv = __hip_atomic_load(tmo, __ATOMIC_RELAXED, __HIP_MEMORY_SCOPE_AGENT);
                    unsigned np = 0;
#pragma unroll
                    for (int pr = 0; pr < 8; ++pr) {
                        const unsigned m = ((v[pr][0][0] & v[pr][0][2] & v[pr][1][0] & v[pr][1][2]) & TAG_MASK) == par &&
                                           ((v[pr][0][0] | v[pr][0][2] | v[pr][1][0] | v[pr][1][2]) & TAG_MASK) == par;
                        if (!__all(m)) np |= 1u << pr;
                    }
                    pend = np;
                    if (!pend) break;
                    if (tries == 0) t0 = __builtin_amdgcn_s_memrealtime();
                    if ((unsigned)__builtin_amdgcn_readfirstlane(tv) == call || __builtin_amdgcn_s_memrealtime() - t0 > DEADLINE_TICKS) {
                        __hip_atomic_store(tmo, call, __ATOMIC_RELAXED, __HIP_MEMORY_SCOPE_AGENT);
                        bad = true;
                        break;
                    }
                    __builtin_amdgcn_s_sleep(2);
                }
                if (lane == 0) {
                    if (same) __builtin_amdgcn_raw_buffer_store_b32(seq0 + ps + 1, wsrs, dnoff + (ps % RING) * 4u, 0, 0);
                    else __builtin_amdgcn_raw_buffer_store_b32(seq0 + ps + 1, wsrs, dnoff + (ps % RING) * 4u, 0, 16);
                }
                f4 o[2];
#pragma unroll
                for (int q = 0; q < 2; ++q) {
                    o[q] = __builtin_bit_cast(f4, v[0][q]);
#pragma unroll
                    for (int pr = 1; pr < 8; ++pr) o[q] += __builtin_bit_cast(f4, v[pr][q]);
                }
                if (ps >= ORING) lds_wait_ge(&lf[F_OFREE], ps - ORING + 1);
                f4 r[2];
#pragma unroll
                for (int q = 0; q < 2; ++q) r[q] = (f4){silu(o[q][0]), silu(o[q][1]), silu(o[q][2]), silu(o[q][3])};
                bad = bad || ((unsigned)__builtin_amdgcn_readfirstlane(tv) == call);
#pragma unroll
                for (int q = 0; q < 2; ++q) {
                    if (bad) r[q] = (f4){__builtin_nanf(""), __builtin_nanf(""), __builtin_nanf(""), __builtin_nanf("")};
                    *(f4*)(smem + LDS_OUT + (ps % ORING) * 2048 + q * 1024 + lane * 16) = r[q];
                }
                asm volatile("s_waitcnt lgkmcnt(0)" ::: "memory");
                if (lane == 0) __hip_atomic_store(&lf[F_ORDY + ps], 1u, __ATOMIC_RELAXED, __HIP_MEMORY_SCOPE_WORKGROUP);
            }
        } else {
            lds_wait_ge(&lf[F_CALL], 1);
            const unsigned call = __builtin_amdgcn_readfirstlane(lds_load(&lf[F_CALL]));
            float* obase = p.out[dest] + (size_t)(rank * ML + kq * 8 + ((lane >> 4) & 1) * 4 + (lane >> 5)) * NB + nb * 128 + c * 4;
            for (int ps = 0; ps < NPASS; ++ps) {
                lds_wait_ge(&lf[F_ORDY + ps], 1);
                f4 r[2];
#pragma unroll
                for (int q = 0; q < 2; ++q) r[q] = *(const f4*)(smem + LDS_OUT + (ps % ORING) * 2048 + q * 1024 + lane * 16);
                asm volatile("s_waitcnt lgkmcnt(0)" ::: "memory");
                if (lane == 0) __hip_atomic_store(&lf[F_OFREE], (unsigned)(ps + 1), __ATOMIC_RELAXED, __HIP_MEMORY_SCOPE_WORKGROUP);
                float* o = obase + (size_t)(pass_m(ps) * 64) * NB + pass_h(ps) * 64;
#pragma unroll
                for (int q = 0; q < 2; ++q)
                    asm volatile("global_store_dwordx4 %0, %1, off sc0 sc1\n\ts_nop 1" :: "v"(o + q * 2 * NB), "v"(r[q]) : "memory");
            }
            asm volatile("s_waitcnt vmcnt(0)" ::: "memory");
            if (lane == 0) {
                if (dest != rank) {
                    __hip_atomic_fetch_add((unsigned*)(p.ws[dest] + OFF_RFLAG + rank * 128), 1u, __ATOMIC_RELAXED, __HIP_MEMORY_SCOPE_SYSTEM);
                } else if (idx < NR && idx != rank) {
                    unsigned* flag = (unsigned*)(myws + OFF_RFLAG + idx * 128);
                    const unsigned target = call * 32u;
                    unsigned long long t0 = __builtin_amdgcn_s_memrealtime();
                    while (__hip_atomic_load(flag, __ATOMIC_RELAXED, __HIP_MEMORY_SCOPE_SYSTEM) < target) {
                        if (__hip_atomic_load(tmo, __ATOMIC_RELAXED, __HIP_MEMORY_SCOPE_AGENT) == call ||
                            __builtin_amdgcn_s_memrealtime() - t0 > DEADLINE_TICKS) {
                            __hip_atomic_store(tmo, call, __ATOMIC_RELAXED, __HIP_MEMORY_SCOPE_AGENT);
                            obase[0] = __builtin_nanf("");
                            break;
                        }
                        __builtin_amdgcn_s_sleep(2);
                    }
                    __builtin_amdgcn_fence(__ATOMIC_ACQUIRE, "");
                }
            }
        }
    }
}

extern "C" void kernel_launch(int rank, int nranks, ncclComm_t comm,
                              hipStream_t stream,
                              void* const* d_in, const int* in_sizes, int n_in,
                              void* d_out, int out_size,
                              const int* in_dtypes, int out_dtype,
                              void* const* d_out_all,
                              void* const* d_ws_all, size_t ws_bytes) {
    if (nranks != NR || ws_bytes < WS_USED || n_in < 2 || rank < 0 || rank >= NR) { fprintf(stderr, "bad config\n"); return; }
    Params p;
    p.x = (const float*)d_in[0];
    p.w = (const float*)d_in[1];
    for (int i = 0; i < NR; ++i) { p.out[i] = (float*)d_out_all[i]; p.ws[i] = (unsigned char*)d_ws_all[i]; }
    p.rank = rank;
    fused_kernel<<<NWG, NTH, 0, stream>>>(p);
    HIP_CHECK(hipGetLastError());
}
```

```cpp
#include <hip/hip_runtime.h>
#include <rccl/rccl.h>
#include <cstdio>
#include <cstdint>

#define HIP_CHECK(expr) do { hipError_t _e = (expr); if (_e != hipSuccess) { \
    fprintf(stderr, "HIP error %s at %s:%d\n", hipGetErrorString(_e), __FILE__, __LINE__); \
    return; } } while (0)

typedef _Float16 h8 __attribute__((ext_vector_type(8)));
typedef float f4 __attribute__((ext_vector_type(4)));
typedef unsigned u4 __attribute__((ext_vector_type(4)));

constexpr int APOL = 16;
constexpr int RPOL = 16;
constexpr int WPOL = 2;
constexpr int XPOL = 2;
constexpr int NWB = 3;
constexpr unsigned long long AT = 400;
constexpr int DVS = 8;

constexpr int NR = 8;
constexpr int ML = 512;
constexpr int KK = 4096;
constexpr int NN = 4096;
constexpr int NB = 512;
constexpr int NWG = 256;
constexpr int NTH = 512;
constexpr int NPASS = 16;
constexpr int RING = 2;
constexpr int ORING = 8;
constexpr unsigned SLOT_BYTES = 4u << 20;
constexpr unsigned long long DEADLINE_TICKS = 50000000ull;
constexpr unsigned TAG_MASK = 7u;
__device__ __forceinline__ unsigned tag_of(int ps, unsigned call) {
    const int r = ps % RING;
    const unsigned uses = (unsigned)((NPASS - 1 - r) / RING + 1);
    const unsigned n = (call - 1) * uses + (unsigned)(ps / RING) + 1u;
    return (n % 7u) + 1u;
}

constexpr unsigned OFF_TMO = 128;
constexpr unsigned OFF_RFLAG = 256;
constexpr unsigned OFF_CTR = 3072;
constexpr unsigned OFF_XCC = 4096;
constexpr unsigned OFF_DONE = 8192;
constexpr unsigned OFF_X16 = 1u << 20;
constexpr unsigned OFF_PART = 9u << 20;
constexpr size_t WS_USED = (size_t)OFF_PART + (size_t)RING * SLOT_BYTES;

constexpr int LDS_W = 0;
constexpr int LDS_OUT = 131072;
constexpr int LDS_FLAG = LDS_OUT + ORING * 2048;
constexpr int LDS_BYTES = LDS_FLAG + 256;
enum { F_CALL = 0, F_WA, F_WB, F_AISS, F_CDONE, F_RTICKET, F_OFREE, F_SAME, F_ORDY  , F_NWORDS = F_ORDY + 16 };

struct Params {
    const float* x;
    const float* w;
    float* out[NR];
    unsigned char* ws[NR];
    int rank;
};

__device__ __forceinline__ float silu(float y) {
    float c = fminf(fmaxf(y, -60.0f), 60.0f);
    return y * __builtin_amdgcn_rcpf(1.0f + __expf(-c));
}

typedef __attribute__((address_space(3))) unsigned lds_u32;
__device__ __forceinline__ unsigned lds_load(lds_u32* p) { return __hip_atomic_load(p, __ATOMIC_RELAXED, __HIP_MEMORY_SCOPE_WORKGROUP); }
__device__ __forceinline__ void lds_add1(lds_u32* p, int lane) {
    asm volatile("s_waitcnt lgkmcnt(0)" ::: "memory");
    if (lane == 0) __hip_atomic_fetch_add(p, 1u, __ATOMIC_RELAXED, __HIP_MEMORY_SCOPE_WORKGROUP);
}

__device__ __forceinline__ unsigned lds_ticket(lds_u32* p) {
    unsigned old;
    unsigned long long save;
    const unsigned addr = (unsigned)(size_t)p, one = 1u;
    asm volatile("s_mov_b64 %1, exec\n\ts_mov_b64 exec, 1\n\tds_add_rtn_u32 %0, %2, %3\n\ts_waitcnt lgkmcnt(0)\n\ts_mov_b64 exec, %1"
                 : "=&v"(old), "=&s"(save) : "v"(addr), "v"(one) : "memory");
    return __builtin_amdgcn_readfirstlane(old);
}

__device__ __forceinline__ bool lds_wait_ge(lds_u32* p, unsigned target) {
    if ((unsigned)__builtin_amdgcn_readfirstlane(lds_load(p)) >= target) return true;
    unsigned long long t0 = __builtin_amdgcn_s_memrealtime();
    while ((unsigned)__builtin_amdgcn_readfirstlane(lds_load(p)) < target) {
        if (__builtin_amdgcn_s_memrealtime() - t0 > 2 * DEADLINE_TICKS) return false;
        __builtin_amdgcn_s_sleep(1);
    }
    return true;
}

__device__ __forceinline__ bool give_up(unsigned long long t0, unsigned* tmo, unsigned call) {
    if ((unsigned)__builtin_amdgcn_readfirstlane(__hip_atomic_load(tmo, __ATOMIC_RELAXED, __HIP_MEMORY_SCOPE_AGENT)) == call ||
        __builtin_amdgcn_s_memrealtime() - t0 > DEADLINE_TICKS) {
        __hip_atomic_store(tmo, call, __ATOMIC_RELAXED, __HIP_MEMORY_SCOPE_AGENT);
        return true;
    }
    return false;
}

__device__ __forceinline__ bool gwait_all_ge(unsigned* word, bool sel, unsigned target, unsigned* tmo, unsigned call) {
    unsigned long long t0 = __builtin_amdgcn_s_memrealtime();
    for (;;) {
        const unsigned v = __hip_atomic_load(word, __ATOMIC_RELAXED, __HIP_MEMORY_SCOPE_AGENT);
        if (__all(!sel || v >= target)) return true;
        if (give_up(t0, tmo, call)) return false;
        __builtin_amdgcn_s_sleep(1);
    }
}

#define RSRC(ptr, bytes) __builtin_amdgcn_make_buffer_rsrc((void*)(ptr), 0, (bytes), 0x00020000)
#define BLOAD(rs, voff, soff, aux) __builtin_amdgcn_raw_buffer_load_b128(rs, voff, soff, aux)

__device__ __forceinline__ h8 cvt8(f4 a, f4 b) {
    h8 r = {(_Float16)a[0], (_Float16)a[1], (_Float16)a[2], (_Float16)a[3], (_Float16)b[0], (_Float16)b[1], (_Float16)b[2], (_Float16)b[3]};
    return r;
}

template <int NU, int WIN, class F>
__device__ __forceinline__ void load_w_units(__amdgpu_buffer_rsrc_t wrs, unsigned voff, unsigned soff0, unsigned char* ldsw, int lane, F after_issue) {
    f4 t[WIN][8];
#define W_ISSUE(u) _Pragma("unroll") for (int jj = 0; jj < 8; ++jj) \
        t[(u) % WIN][jj] = __builtin_bit_cast(f4, BLOAD(wrs, voff, soff0 + (unsigned)((u) * 32 + (jj & 3) + 16 * (jj >> 2)) * (NN * 4u), WPOL))
#pragma unroll
    for (int u = 0; u < WIN; ++u) { W_ISSUE(u); }
    __builtin_amdgcn_sched_barrier(0);
    after_issue();
    __builtin_amdgcn_sched_barrier(0);
#pragma unroll
    for (int u = 0; u < NU; ++u) {
        h8 f[4];
#pragma unroll
        for (int tt = 0; tt < 4; ++tt)
#pragma unroll
            for (int jj = 0; jj < 8; ++jj) f[tt][jj] = (_Float16)t[u % WIN][jj][tt];
        __builtin_amdgcn_sched_barrier(0);
        if (u + WIN < NU) { W_ISSUE(u + WIN); }
#pragma unroll
        for (int tt = 0; tt < 4; ++tt) *(h8*)(ldsw + (u * 8 + tt) * 1024 + lane * 16) = f[tt];
        __builtin_amdgcn_sched_barrier(0);
    }
#undef W_ISSUE
}

__device__ __forceinline__ int pass_m(int q) { return q < 2 ? q : (q < 14 ? 2 + ((q - 2) >> 1) : 15 - q); }
__device__ __forceinline__ int pass_h(int q) { return q < 2 ? 0 : (q < 14 ? (q & 1) : 1); }

typedef __attribute__((address_space(3))) const h8 lds_h8;
__device__ __forceinline__ h8 lds_frag(unsigned addr, int imm) { return *(lds_h8*)(size_t)(addr + imm); }

struct Fin {
    __amdgpu_buffer_rsrc_t wsrs;
    unsigned char* myws;
    unsigned* tmo;
    lds_u32* cdone;
    unsigned pvoff, dvoff0, seq0, call;
    int lane;
    bool same;
};

__device__ __forceinline__ void ensure_a(h8 (&a)[16], __amdgpu_buffer_rsrc_t wsrs, unsigned avoff, unsigned soff, unsigned* tmo, unsigned call) {
    unsigned long long t0 = 0;
    for (int tries = 0;; ++tries) {
        unsigned m = ~0u;
#pragma unroll
        for (int s = 0; s < 16; ++s) { const u4 w = __builtin_bit_cast(u4, a[s]); m = min(m, min(w[0], w[2])); }
        if (__all(m != 0u)) return;
        if (tries == 0) t0 = __builtin_amdgcn_s_memrealtime();
        if (give_up(t0, tmo, call)) return;
        asm volatile("" ::: "memory");
#pragma unroll
        for (int s = 0; s < 16; ++s) a[s] = __builtin_bit_cast(h8, BLOAD(wsrs, avoff, soff + s * 1024, 16));
    }
}

__device__ __forceinline__ void publish(const Fin& f, int ps, const f4 (&acc)[4], unsigned dv) {
    if (ps >= RING) {
        const unsigned need = f.seq0 + (ps - RING + 1);
        if (!__all(dv >= need)) gwait_all_ge((unsigned*)(f.myws + f.dvoff0 + (ps % RING) * 4u), true, need, f.tmo, f.call);
    }
    const unsigned so = (unsigned)(ps % RING) * SLOT_BYTES;
    const unsigned par = tag_of(ps, f.call);
#pragma unroll
    for (int jr = 0; jr < 4; ++jr) {
        u4 v = {(__float_as_uint(acc[0][jr]) & ~TAG_MASK) | par, __float_as_uint(acc[1][jr]), (__float_as_uint(acc[2][jr]) & ~TAG_MASK) | par, __float_as_uint(acc[3][jr])};
        if (f.same) __builtin_amdgcn_raw_buffer_store_b128(v, f.wsrs, f.pvoff, so + jr * 512, 0);
        else __builtin_amdgcn_raw_buffer_store_b128(v, f.wsrs, f.pvoff, so + jr * 512, 16);
    }
    if (f.lane == 0) __hip_atomic_fetch_add(f.cdone, 1u, __ATOMIC_RELAXED, __HIP_MEMORY_SCOPE_WORKGROUP);
}

template <bool RELOAD, bool PREV>
__device__ __forceinline__ void pass_body(int ps, h8 (&a)[16], f4 (&acc)[4], const f4 (&accp)[4], unsigned dvp, unsigned& dv, h8 (&b)[3][4],
                                          unsigned wb, unsigned wb_next, const Fin& f, unsigned avoff, unsigned nsoff, unsigned csoff) {
    unsigned amin = ~0u;
    unsigned lo = wb, hi = wb + 65536u;
    asm volatile("" : "+v"(lo), "+v"(hi));
#pragma unroll
    for (int s = 0; s < 16; ++s) {
#pragma unroll
        for (int t = 0; t < 4; ++t) {
            if (s == 0) acc[t] = __builtin_amdgcn_mfma_f32_16x16x32_f16(a[s], b[0][t], (f4){0.f, 0.f, 0.f, 0.f}, 0, 0, 0);
            else acc[t] = __builtin_amdgcn_mfma_f32_16x16x32_f16(a[s], b[s % 3][t], acc[t], 0, 0, 0);
            if (s + 2 < 8) b[(s + 2) % 3][t] = lds_frag(lo, ((s + 2) * 8 + t) * 1024);
            else if (s + 2 < 16) b[(s + 2) % 3][t] = lds_frag(hi, ((s + 2 - 8) * 8 + t) * 1024);
            __builtin_amdgcn_sched_barrier(0);
        }
        if (s == 0 && PREV) publish(f, ps - 1, accp, dvp);
        if (s == DVS) {
            asm volatile("" ::: "memory");
            dv = __builtin_amdgcn_raw_buffer_load_b32(f.wsrs, f.dvoff0, (ps % RING) * 4u, 16);
            asm volatile("" ::: "memory");
        }
        { const u4 w = __builtin_bit_cast(u4, a[s]); amin = min(amin, min(w[0], w[2])); }
        if (RELOAD) a[s] = __builtin_bit_cast(h8, BLOAD(f.wsrs, avoff, nsoff + s * 1024, APOL));
        __builtin_amdgcn_sched_barrier(0);
    }
    if (!__all(amin != 0u)) {
#pragma unroll
        for (int t = 0; t < 4; ++t) acc[t] = (f4){0.f, 0.f, 0.f, 0.f};
        unsigned long long t0 = __builtin_amdgcn_s_memrealtime();
#pragma unroll 1
        for (int q = 0; q < 4; ++q) {
            u4 w[4];
            for (;;) {
                asm volatile("" ::: "memory");
                unsigned m = ~0u;
#pragma unroll
                for (int i = 0; i < 4; ++i) w[i] = BLOAD(f.wsrs, avoff, csoff + (q * 4 + i) * 1024, 16);
#pragma unroll
                for (int i = 0; i < 4; ++i) m = min(m, min(w[i][0], w[i][2]));
                if (__all(m != 0u)) break;
                if (give_up(t0, f.tmo, f.call)) break;
            }
#pragma unroll
            for (int i = 0; i < 4; ++i)
#pragma unroll
                for (int t = 0; t < 4; ++t)
                    acc[t] = __builtin_amdgcn_mfma_f32_16x16x32_f16(__builtin_bit_cast(h8, w[i]), lds_frag(wb + (q * 4 + i) * 8192u, t * 1024), acc[t], 0, 0, 0);
        }
    }
    unsigned nx = wb_next;
    asm volatile("" : "+v"(nx));
#pragma unroll
    for (int s = 0; s < 2; ++s)
#pragma unroll
        for (int t = 0; t < 4; ++t) b[s][t] = lds_frag(nx, (s * 8 + t) * 1024);
}

template <class F>
__device__ __forceinline__ void prepass(const Params& p, __amdgpu_buffer_rsrc_t wsrs, lds_u32* lf, int idx, int j, int q4, int lane, F after_issue) {
    const int c = lane & 15, g = lane >> 4;
    const auto xrs = RSRC(p.x, ML * KK * 4u);
    const unsigned voff = ((unsigned)(idx * 16 + c) * KK + j * 512 + q4 * 128 + g * 4) * 4u;
    f4 t[8];
#pragma unroll
    for (int q = 0; q < 4; ++q) {
        t[2 * q] = __builtin_bit_cast(f4, BLOAD(xrs, voff + q * 128, 0, XPOL));
        t[2 * q + 1] = __builtin_bit_cast(f4, BLOAD(xrs, voff + q * 128 + 64, 0, XPOL));
    }
    __builtin_amdgcn_sched_barrier(0);
    after_issue();
    __builtin_amdgcn_sched_barrier(0);
    u4 f[4];
#pragma unroll
    for (int q = 0; q < 4; ++q) {
        f[q] = __builtin_bit_cast(u4, cvt8(t[2 * q], t[2 * q + 1]));
        if (f[q][0] == 0u) f[q][0] = 0x8000u;
        if (f[q][2] == 0u) f[q][2] = 0x8000u;
    }
    lds_wait_ge(&lf[F_CALL], 1);
    const unsigned call = __builtin_amdgcn_readfirstlane(lds_load(&lf[F_CALL]));
    const unsigned xo = OFF_X16 + (call & 1u) * (4u << 20) + (unsigned)((idx * 8 + j) * 16 + q4 * 4) * 1024u + lane * 16u;
#pragma unroll
    for (int q = 0; q < 4; ++q) __builtin_amdgcn_raw_buffer_store_b128(f[q], wsrs, xo + q * 1024, 0, 16);
}

__global__ __launch_bounds__(NTH, 2) void fused_kernel(Params p) {
    const unsigned long long tstart = __builtin_amdgcn_s_memrealtime();
    __shared__ __attribute__((aligned(16))) unsigned char smem[LDS_BYTES];
    const int tid = threadIdx.x, lane = tid & 63;
    const int wave = __builtin_amdgcn_readfirstlane(tid >> 6);
    const int bid = blockIdx.x;
    const int rank = p.rank;
    const int j = bid & 7, idx = bid >> 3;
    const int dest = j;
    const int kq = idx & 7, nb = idx >> 3;
    const int grp = j * 4 + nb;
    unsigned char* myws = p.ws[rank];
    lds_u32* lf = (lds_u32*)(smem + LDS_FLAG);
    unsigned ctr = 0;
    if (wave == 0) ctr = __hip_atomic_load((unsigned*)(myws + OFF_CTR) + bid, __ATOMIC_RELAXED, __HIP_MEMORY_SCOPE_AGENT);
    if (tid < F_NWORDS) lf[tid] = 0;
    asm volatile("s_waitcnt lgkmcnt(0)" ::: "memory");
    __builtin_amdgcn_s_barrier();
    unsigned* tmo = (unsigned*)(myws + OFF_TMO);
    const auto wsrs = RSRC(myws, (unsigned)WS_USED);
    const int c = lane & 15, g = lane >> 4;

    if (wave < 4) {
        prepass(p, wsrs, lf, idx, j, wave, lane, [&]() {
            if (wave == 0) {
                const unsigned call0 = __builtin_amdgcn_readfirstlane(ctr) + 1;
                if (lane == 0) {
                    __hip_atomic_store(&lf[F_CALL], call0, __ATOMIC_RELAXED, __HIP_MEMORY_SCOPE_WORKGROUP);
                    unsigned xcc;
                    asm volatile("s_getreg_b32 %0, hwreg(HW_REG_XCC_ID, 0, 4)" : "=s"(xcc));
                    __hip_atomic_store((unsigned*)(myws + OFF_XCC + grp * 128 + kq * 4), (call0 << 4) | xcc, __ATOMIC_RELAXED, __HIP_MEMORY_SCOPE_AGENT);
                    __hip_atomic_store((unsigned*)(myws + OFF_CTR) + bid, call0, __ATOMIC_RELAXED, __HIP_MEMORY_SCOPE_AGENT);
                }
            }
        });
        const unsigned call = __builtin_amdgcn_readfirstlane(lds_load(&lf[F_CALL]));
        const unsigned avoff = OFF_X16 + (call & 1u) * (4u << 20) + (unsigned)((wave * 8 + kq) * 16) * 1024u + lane * 16u;
        h8 a[2][16];
        unsigned xcc_v;
        {
            asm volatile("s_waitcnt vmcnt(0)" ::: "memory");
            while (__builtin_amdgcn_s_memrealtime() - tstart < AT) __builtin_amdgcn_s_sleep(1);
            unsigned* tab = (unsigned*)(myws + OFF_XCC + grp * 128) + (lane & 7);
            xcc_v = __hip_atomic_load(tab, __ATOMIC_RELAXED, __HIP_MEMORY_SCOPE_AGENT);
#pragma unroll
            for (int q = 0; q < 2; ++q)
#pragma unroll
                for (int s = 0; s < 16; ++s) a[q][s] = __builtin_bit_cast(h8, BLOAD(wsrs, avoff, q * (512u << 10) + s * 1024, 16));
            if (!__all((xcc_v >> 4) >= call)) {
                gwait_all_ge(tab, true, call << 4, tmo, call);
                xcc_v = __hip_atomic_load(tab, __ATOMIC_RELAXED, __HIP_MEMORY_SCOPE_AGENT);
            }
            ensure_a(a[0], wsrs, avoff, 0u, tmo, call);
            ensure_a(a[1], wsrs, avoff, 512u << 10, tmo, call);
            if (lane == 0) __hip_atomic_fetch_add(&lf[F_AISS], 1u, __ATOMIC_RELAXED, __HIP_MEMORY_SCOPE_WORKGROUP);
        }
        const unsigned wb0 = (unsigned)(size_t)(lds_h8*)(smem + LDS_W) + lane * 16;
        Fin f;
        f.wsrs = wsrs; f.myws = myws; f.tmo = tmo; f.cdone = &lf[F_CDONE]; f.call = call; f.lane = lane;
        f.dvoff0 = OFF_DONE + (unsigned)(grp * 8 + 2 * wave + (lane & 1)) * 128u;
        f.seq0 = (call - 1) * NPASS;
        f.pvoff = OFF_PART + (unsigned)((grp * 8 + 2 * wave + (lane >> 5)) * 8 + kq) * 2048u + (lane & 31) * 16u;
        f.same = __all(xcc_v == (unsigned)__builtin_amdgcn_readfirstlane(xcc_v));
        if (wave == 0 && lane == 0) __hip_atomic_store(&lf[F_SAME], f.same ? 1u : 2u, __ATOMIC_RELAXED, __HIP_MEMORY_SCOPE_WORKGROUP);
        lds_wait_ge(&lf[F_WA], 4);
        f4 acc[2][4];
        unsigned dv[2] = {0u, 0u};
        h8 b[3][4];
#pragma unroll
        for (int s = 0; s < 2; ++s)
#pragma unroll
            for (int t = 0; t < 4; ++t) b[s][t] = lds_frag(wb0, (s * 8 + t) * 1024);
        auto bo = [&](int m) { return (unsigned)m * (512u << 10); };
        const unsigned wbA = wb0, wbB = wb0 + 4096;
        pass_body<false, false>(0, a[0], acc[0], acc[1], 0u, dv[0], b, wbA, wbA, f, avoff, 0u, bo(0));
        publish(f, 0, acc[0], dv[0]);
        pass_body<false, false>(1, a[1], acc[1], acc[0], 0u, dv[1], b, wbA, wbA, f, avoff, 0u, bo(1));
        publish(f, 1, acc[1], dv[1]);
#pragma unroll
        for (int q = 0; q < 2; ++q)
#pragma unroll
            for (int s = 0; s < 16; ++s) a[q][s] = __builtin_bit_cast(h8, BLOAD(wsrs, avoff, bo(2 + q) + s * 1024, APOL));
        lds_wait_ge(&lf[F_WB], NWB);
        pass_body<false, false>(2, a[0], acc[0], acc[1], 0u, dv[0], b, wbA, wbB, f, avoff, 0u, bo(2));
        pass_body<true, true>(3, a[0], acc[1], acc[0], dv[0], dv[1], b, wbB, wbA, f, avoff, bo(4), bo(2));
        pass_body<false, true>(4, a[1], acc[0], acc[1], dv[1], dv[0], b, wbA, wbB, f, avoff, 0u, bo(3));
        pass_body<true, true>(5, a[1], acc[1], acc[0], dv[0], dv[1], b, wbB, wbA, f, avoff, bo(5), bo(3));
        for (int m = 4; m < 8; m += 2) {
            const int ps = 2 * m - 2;
            pass_body<false, true>(ps, a[0], acc[0], acc[1], dv[1], dv[0], b, wbA, wbB, f, avoff, 0u, bo(m));
            pass_body<true, true>(ps + 1, a[0], acc[1], acc[0], dv[0], dv[1], b, wbB, wbA, f, avoff, bo(m == 4 ? 6 : 1), bo(m));
            pass_body<false, true>(ps + 2, a[1], acc[0], acc[1], dv[1], dv[0], b, wbA, wbB, f, avoff, 0u, bo(m + 1));
            pass_body<true, true>(ps + 3, a[1], acc[1], acc[0], dv[0], dv[1], b, wbB, m == 4 ? wbA : wbB, f, avoff, bo(m == 4 ? 7 : 0), bo(m + 1));
        }
        pass_body<false, true>(14, a[0], acc[0], acc[1], dv[1], dv[0], b, wbB, wbB, f, avoff, 0u, bo(1));
        pass_body<false, true>(15, a[1], acc[1], acc[0], dv[0], dv[1], b, wbB, wbB, f, avoff, 0u, bo(0));
        publish(f, NPASS - 1, acc[1], dv[1]);
        {
            const unsigned xz = OFF_X16 + ((call + 1) & 1u) * (4u << 20) + (unsigned)((idx * 8 + j) * 16 + wave * 4) * 1024u + lane * 16u;
            const u4 z = {0u, 0u, 0u, 0u};
#pragma unroll
            for (int q = 0; q < 4; ++q) __builtin_amdgcn_raw_buffer_store_b128(z, wsrs, xz + q * 1024, 0, 16);
        }
    } else {
        const auto wrs = RSRC(p.w, (unsigned)KK * NN * 4u);
        const unsigned wvoff = ((unsigned)(kq * 512 + g * 4) * NN + dest * NB + nb * 128 + c * 4) * 4u;
        {
            const int u = wave - 4;
            load_w_units<4, 4>(wrs, wvoff, (unsigned)(u * 128) * (NN * 4u), smem + LDS_W + u * 32768, lane, []() {});
            lds_add1(&lf[F_WA], lane);
        }
        if (wave != 6) {
            lds_wait_ge(&lf[F_AISS], 4);
            if (wave == 5) load_w_units<6, 6>(wrs, wvoff + 256, 0u, smem + LDS_W + 4096, lane, []() {});
            else {
                const int s0 = wave == 4 ? 6 : 11;
                load_w_units<5, 5>(wrs, wvoff + 256, (unsigned)(s0 * 32) * (NN * 4u), smem + LDS_W + 4096 + s0 * 8192, lane, []() {});
            }
            lds_add1(&lf[F_WB], lane);
        }
        if (wave < 7) {
            lds_wait_ge(&lf[F_CALL], 1);
            const unsigned call = __builtin_amdgcn_readfirstlane(lds_load(&lf[F_CALL]));
            lds_wait_ge(&lf[F_SAME], 1);
            const bool same = __builtin_amdgcn_readfirstlane(lds_load(&lf[F_SAME])) == 1;
            const unsigned dnoff = OFF_DONE + (unsigned)(grp * 8 + kq) * 128u;
            const unsigned seq0 = (call - 1) * NPASS;
            const unsigned pb0 = OFF_PART + (unsigned)((grp * 8 + kq) * 8) * 2048u + lane * 16u;
            for (;;) {
                const int ps = (int)lds_ticket(&lf[F_RTICKET]);
                if (ps >= NPASS) break;
                lds_wait_ge(&lf[F_CDONE], 4 * (ps + 1));
                const unsigned so = (unsigned)(ps % RING) * SLOT_BYTES;
                const unsigned par = tag_of(ps, call);
                u4 v[8][2];
                bool bad = false;
                unsigned long long t0 = 0;
                unsigned pend = 0xffu;
                unsigned tv = 0;
                for (int tries = 0;; ++tries) {
#pragma unroll
                    for (int pr = 0; pr < 8; ++pr)
                        if (pend & (1u << pr)) {
                            asm volatile("" ::: "memory");
                            v[pr][0] = BLOAD(wsrs, pb0, so + pr * 2048, RPOL);
                            v[pr][1] = BLOAD(wsrs, pb0 + 1024, so + pr * 2048, RPOL);
                        }
                    tv = __hip_atomic_load(tmo, __ATOMIC_RELAXED, __HIP_MEMORY_SCOPE_AGENT);
                    unsigned np = 0;
#pragma unroll
                    for (int pr = 0; pr < 8; ++pr) {
                        const unsigned m = ((v[pr][0][0] & v[pr][0][2] & v[pr][1][0] & v[pr][1][2]) & TAG_MASK) == par &&
                                           ((v[pr][0][0] | v[pr][0][2] | v[pr][1][0] | v[pr][1][2]) & TAG_MASK) == par;
                        if (!__all(m)) np |= 1u << pr;
                    }
                    pend = np;
                    if (!pend) break;
                    if (tries == 0) t0 = __builtin_amdgcn_s_memrealtime();
                    if ((unsigned)__builtin_amdgcn_readfirstlane(tv) == call || __builtin_amdgcn_s_memrealtime() - t0 > DEADLINE_TICKS) {
                        __hip_atomic_store(tmo, call, __ATOMIC_RELAXED, __HIP_MEMORY_SCOPE_AGENT);
                        bad = true;
                        break;
                    }
                    __builtin_amdgcn_s_sleep(2);
                }
                if (lane == 0) {
                    if (same) __builtin_amdgcn_raw_buffer_store_b32(seq0 + ps + 1, wsrs, dnoff + (ps % RING) * 4u, 0, 0);
                    else __builtin_amdgcn_raw_buffer_store_b32(seq0 + ps + 1, wsrs, dnoff + (ps % RING) * 4u, 0, 16);
                }
                f4 o[2];
#pragma unroll
                for (int q = 0; q < 2; ++q) {
                    o[q] = (f4){0.f, 0.f, 0.f, 0.f};
#pragma unroll
                    for (int pr = 0; pr < 8; ++pr) {
                        const u4 u = {v[pr][q][0] & ~TAG_MASK, v[pr][q][1], v[pr][q][2] & ~TAG_MASK, v[pr][q][3]};
                        o[q] += __builtin_bit_cast(f4, u);
                    }
                }
                if (ps >= ORING) lds_wait_ge(&lf[F_OFREE], ps - ORING + 1);
                f4 r[2];
#pragma unroll
                for (int q = 0; q < 2; ++q) r[q] = (f4){silu(o[q][0]), silu(o[q][1]), silu(o[q][2]), silu(o[q][3])};
                bad = bad || ((unsigned)__builtin_amdgcn_readfirstlane(tv) == call);
#pragma unroll
                for (int q = 0; q < 2; ++q) {
                    if (bad) r[q] = (f4){__builtin_nanf(""), __builtin_nanf(""), __builtin_nanf(""), __builtin_nanf("")};
                    *(f4*)(smem + LDS_OUT + (ps % ORING) * 2048 + q * 1024 + lane * 16) = r[q];
                }
                asm volatile("s_waitcnt lgkmcnt(0)" ::: "memory");
                if (lane == 0) __hip_atomic_store(&lf[F_ORDY + ps], 1u, __ATOMIC_RELAXED, __HIP_MEMORY_SCOPE_WORKGROUP);
            }
        } else {
            lds_wait_ge(&lf[F_CALL], 1);
            const unsigned call = __builtin_amdgcn_readfirstlane(lds_load(&lf[F_CALL]));
            float* obase = p.out[dest] + (size_t)(rank * ML + kq * 8 + ((lane >> 4) & 1) * 4 + (lane >> 5)) * NB + nb * 128 + c * 4;
            for (int ps = 0; ps < NPASS; ++ps) {
                lds_wait_ge(&lf[F_ORDY + ps], 1);
                f4 r[2];
#pragma unroll
                for (int q = 0; q < 2; ++q) r[q] = *(const f4*)(smem + LDS_OUT + (ps % ORING) * 2048 + q * 1024 + lane * 16);
                asm volatile("s_waitcnt lgkmcnt(0)" ::: "memory");
                if (lane == 0) __hip_atomic_store(&lf[F_OFREE], (unsigned)(ps + 1), __ATOMIC_RELAXED, __HIP_MEMORY_SCOPE_WORKGROUP);
                float* o = obase + (size_t)(pass_m(ps) * 64) * NB + pass_h(ps) * 64;
#pragma unroll
                for (int q = 0; q < 2; ++q)
                    asm volatile("global_store_dwordx4 %0, %1, off sc0 sc1\n\ts_nop 1" :: "v"(o + q * 2 * NB), "v"(r[q]) : "memory");
            }
            asm volatile("s_waitcnt vmcnt(0)" ::: "memory");
            if (lane == 0) {
                if (dest != rank) {
                    __hip_atomic_fetch_add((unsigned*)(p.ws[dest] + OFF_RFLAG + rank * 128), 1u, __ATOMIC_RELAXED, __HIP_MEMORY_SCOPE_SYSTEM);
                } else if (idx < NR && idx != rank) {
                    unsigned* flag = (unsigned*)(myws + OFF_RFLAG + idx * 128);
                    const unsigned target = call * 32u;
                    unsigned long long t0 = __builtin_amdgcn_s_memrealtime();
                    while (__hip_atomic_load(flag, __ATOMIC_RELAXED, __HIP_MEMORY_SCOPE_SYSTEM) < target) {
                        if (__hip_atomic_load(tmo, __ATOMIC_RELAXED, __HIP_MEMORY_SCOPE_AGENT) == call ||
                            __builtin_amdgcn_s_memrealtime() - t0 > DEADLINE_TICKS) {
                            __hip_atomic_store(tmo, call, __ATOMIC_RELAXED, __HIP_MEMORY_SCOPE_AGENT);
                            obase[0] = __builtin_nanf("");
                            break;
                        }
                        __builtin_amdgcn_s_sleep(2);
                    }
                    __builtin_amdgcn_fence(__ATOMIC_ACQUIRE, "");
                }
            }
        }
    }
}

extern "C" void kernel_launch(int rank, int nranks, ncclComm_t comm,
                              hipStream_t stream,
                              void* const* d_in, const int* in_sizes, int n_in,
                              void* d_out, int out_size,
                              const int* in_dtypes, int out_dtype,
                              void* const* d_out_all,
                              void* const* d_ws_all, size_t ws_bytes) {
    if (nranks != NR || ws_bytes < WS_USED || n_in < 2 || rank < 0 || rank >= NR) { fprintf(stderr, "bad config\n"); return; }
    Params p;
    p.x = (const float*)d_in[0];
    p.w = (const float*)d_in[1];
    for (int i = 0; i < NR; ++i) { p.out[i] = (float*)d_out_all[i]; p.ws[i] = (unsigned char*)d_ws_all[i]; }
    p.rank = rank;
    fused_kernel<<<NWG, NTH, 0, stream>>>(p);
    HIP_CHECK(hipGetLastError());
}
```
